```python
import numpy as np

kernel_name = "gpu_ag_gemm_m4096_k4096_n32768_bf16_gelu"


ABSMAX_RTOL = 0.008

COMPUTE_DTYPE = "bfloat16"
EPILOGUE = "gelu"
IN_QUANT = None
OUT_QUANT = None
IN_DTYPES = {"x": "bfloat16", "w_mat": "bfloat16"}
WORLD_SIZE = 8

def _to_bf16(x):
    u = np.ascontiguousarray(x, dtype=np.float32).view(np.uint32)
    return ((u + 0x7FFF + ((u >> 16) & 1)) & 0xFFFF0000).view(np.float32)

def setup_inputs():

    x = _to_bf16(np.random.randn(4096, 4096).astype(np.float32))
    w_mat = _to_bf16((np.random.randn(4096, 32768) / np.sqrt(4096)).astype(np.float32))
    return {"x": x, "w_mat": w_mat}

def _epilogue(y):
    c = 0.7978845608028654
    return (0.5 * y * (1.0 + np.tanh(c * (y + 0.044715 * y ** 3)))).astype(np.float32)

def reference(x, w_mat):
    y = (x @ w_mat).astype(np.float32)
    return _epilogue(y)
```

```cpp
#include <hip/hip_runtime.h>
#include <rccl/rccl.h>
#include <cstdio>

#define HIP_CHECK(expr) do { hipError_t _e = (expr); if (_e != hipSuccess) { \
    fprintf(stderr, "HIP error %s at %s:%d\n", hipGetErrorString(_e), __FILE__, __LINE__); \
    return; } } while (0)

namespace agg {
#define LAS __attribute__((address_space(3)))
typedef short bf16x8 __attribute__((ext_vector_type(8)));
typedef short s16x4 __attribute__((ext_vector_type(4)));
typedef float f32x4 __attribute__((ext_vector_type(4)));
typedef unsigned u32x4 __attribute__((ext_vector_type(4)));

constexpr int NR = 8;
constexpr int MT = 4096, KT = 4096, NL = 4096;
constexpr int MS = MT / NR;
constexpr int BM = 256, BK = 64, HALF = 128, HTB = 16384, NKT = KT / BK;
constexpr unsigned KSTEP = 32u * HTB;
static_assert(KSTEP == 64u * NL * 2u, "");
constexpr unsigned IMG_BYTES = (unsigned)MT * KT * 2;
constexpr unsigned X_BYTES = (unsigned)MS * KT * 2, W_BYTES = (unsigned)KT * NL * 2;
constexpr size_t WS_CTR = 0;
constexpr size_t WS_TMO = 32768;
constexpr size_t WS_FLAG = 65536;
constexpr unsigned FLAG_BYTES = NR * 8 * 512;
constexpr size_t WS_IMG = 1u << 20;
constexpr size_t WS_NEED = WS_IMG + 2 * (size_t)IMG_BYTES;
constexpr int L_STG = 8 * HTB;
constexpr int L_DEAD = L_STG + 16384;
constexpr int LDS_BYTES = L_DEAD + 16;
constexpr unsigned OOB = 0x40000000u;
#define LEAD 6
#define PACE 110
constexpr unsigned long long DEADLINE = 50000000ull;

struct Args { const char* x; const char* W; char* out; char* ws[NR]; int rank; };

__host__ __device__ __forceinline__ int lds_byte(int r, int c) { const int st = (r >> 4) * 2 + (c >> 5), rr = r & 15, cc = c & 31, ob = rr * 64 + cc * 2; return st * 1024 + (ob ^ (((ob >> 9) & 1) << 5)); }
__host__ __device__ __forceinline__ void stage_rc(int b, int& R, int& C) { const int st = b / 1024, sb = b % 1024, swz = sb ^ (((sb >> 9) & 1) << 5); R = (st >> 1) * 16 + swz / 64; C = (st & 1) * 32 + (swz % 64) / 2; }
__host__ __device__ __forceinline__ int fk(int k) { return (k & 1) | ((k & 2) << 1) | (k & 8); }

__device__ __forceinline__ unsigned cvt_pk_bf16(float lo, float hi) { unsigned r; asm("v_cvt_pk_bf16_f32 %0, %1, %2" : "=v"(r) : "v"(lo), "v"(hi)); return r; }
__device__ __forceinline__ float gelu1(float y) {
  const float k1 = -2.0f * 0.7978845608028654f * 1.4426950408889634f, k2 = k1 * 0.044715f;
  const float z = y * __builtin_fmaf(y * y, k2, k1);
  return y * __builtin_amdgcn_rcpf(1.0f + __builtin_amdgcn_exp2f(z));
}
__device__ __forceinline__ __amdgpu_buffer_rsrc_t mk_rsrc(const void* p, unsigned bytes) { return __builtin_amdgcn_make_buffer_rsrc((void*)p, (short)0, (int)bytes, 0x00020000); }

__device__ __forceinline__ void spin_flags(const unsigned* f, unsigned need, unsigned* tmo, unsigned call, LAS unsigned* dead) {
  const unsigned long long t0 = __builtin_amdgcn_s_memrealtime();
  __builtin_amdgcn_s_waitcnt(0xC07F);
  for (;;) {
    const unsigned v = __hip_atomic_load(f, __ATOMIC_RELAXED, __HIP_MEMORY_SCOPE_SYSTEM);
    if (__all((int)(v - need) >= 0)) break;
    if (__builtin_amdgcn_readfirstlane(__hip_atomic_load(tmo, __ATOMIC_RELAXED, __HIP_MEMORY_SCOPE_AGENT)) == call ||
        __builtin_amdgcn_s_memrealtime() - t0 > DEADLINE) {
      __hip_atomic_store(tmo, call, __ATOMIC_RELAXED, __HIP_MEMORY_SCOPE_AGENT);
      *dead = 1u;
      asm volatile("s_waitcnt vmcnt(0) lgkmcnt(0)" ::: "memory");
      break;
    }
    __builtin_amdgcn_s_sleep(1);
  }
  asm volatile("" ::: "memory");
}

__global__ __launch_bounds__(512, 2) void ag_gemm_kernel(const Args a) {
  extern __shared__ __attribute__((aligned(16))) unsigned char shm_[];
  LAS unsigned char* lds = (LAS unsigned char*)shm_;
  const int tid = threadIdx.x, wid = __builtin_amdgcn_readfirstlane(tid >> 6), lane = tid & 63, wr = wid >> 2, wc = wid & 3, fr = lane & 15, fq = lane >> 4;
  const int bid = blockIdx.x, rank = a.rank;
  char* const ws = a.ws[rank];
  unsigned prev;
  asm volatile("s_load_dword %0, %1, 0x0\n\ts_waitcnt lgkmcnt(0)" : "=s"(prev) : "s"(ws + WS_CTR + (size_t)bid * 128) : "memory");
  const unsigned call = prev + 1u, base = prev * (unsigned)NKT, par = call & 1u;
  const unsigned long long t_start = __builtin_amdgcn_s_memrealtime();
  const int xcd = bid & 7, idx = bid >> 3;
  const int pm = (xcd & 1) * 8 + (idx & 7), pn = (xcd >> 1) * 4 + (idx >> 3);
  const int unit = (bid & 7) | ((bid >> 6) << 3), dest = (rank + ((bid >> 3) & 7)) & (NR - 1);
  char* const wsd = a.ws[dest];
  const __amdgpu_buffer_rsrc_t srdA = mk_rsrc(ws + WS_IMG + (size_t)par * IMG_BYTES, IMG_BYTES);
  const __amdgpu_buffer_rsrc_t srdB = mk_rsrc(a.W, W_BYTES);
  const __amdgpu_buffer_rsrc_t srdX = mk_rsrc(a.x, X_BYTES);
  const __amdgpu_buffer_rsrc_t srdD = mk_rsrc(wsd + WS_IMG + (size_t)par * IMG_BYTES, IMG_BYTES);
  const __amdgpu_buffer_rsrc_t srdFd = mk_rsrc(wsd + WS_FLAG, FLAG_BYTES);
  const __amdgpu_buffer_rsrc_t srdFl = mk_rsrc(ws + WS_FLAG, FLAG_BYTES);
  unsigned* const tmo = (unsigned*)(ws + WS_TMO);
  const unsigned* const myflags = (const unsigned*)(ws + WS_FLAG) + (pm >> 1) * 1024 + (pm & 1) * 64 + (lane & 15) * 4;
  LAS unsigned* const dead = (LAS unsigned*)(lds + L_DEAD);
  const unsigned ldsb = (unsigned)(size_t)lds;

  const unsigned voffA = (unsigned)(wid * 2048 + lane * 16);
  const int kB = wid * 8 + (lane >> 4);
  const unsigned voffB0 = (unsigned)kB * (NL * 2) + (unsigned)(((lane & 15) ^ fk(kB)) * 16);
  const unsigned voffB1 = voffB0 + 4 * (NL * 2) - 1024;
  const unsigned ldsw = ldsb + (unsigned)wid * 2048u;
  const int aoff = lds_byte(wr * 64 + fr, fq * 8);
  const int q_ = (lane & 15) >> 2, p_ = lane & 3;
  const int f_ = (q_ & 1) | ((q_ & 2) << 1) | ((fq & 1) << 3);
  int boff0 = 4 * HTB + 256 * (8 * fq + q_) + 16 * ((4 * wc + p_) ^ f_) + 8 * (p_ & 1);
  int boff1 = boff0 ^ 8;
  asm volatile("" : "+v"(boff0), "+v"(boff1));
  const unsigned sA0 = (unsigned)(2 * pm) * HTB, sB0 = (unsigned)pn * 512u;
  const unsigned sD0 = (unsigned)(rank * 4 + (unit >> 3)) * HTB + (unsigned)(unit & 7) * 2048u;

  unsigned vx1, vx2;
  {
    int R, C; stage_rc((unit & 7) * 2048 + wc * 512 + (lane & 31) * 16, R, C);
    const unsigned src = (unsigned)(((unit >> 3) * 128 + R) * KT + C) * 2u;
    const unsigned dst = (unsigned)(wc * 512 + (lane & 31) * 16);
    if (wr) { vx1 = lane < 32 ? src : OOB; vx2 = lane < 32 ? dst : OOB; }
    else { vx1 = lane < 16 ? (unsigned)((pm >> 1) * 4096 + (pm & 1) * 256 + lane * 16) : OOB; vx2 = (tid == 0) ? (unsigned)(rank * 4096 + unit * 16) : OOB; }
  }
  const __amdgpu_buffer_rsrc_t srdX1 = wr ? srdX : srdFl, srdX2 = wr ? srdD : srdFd;
  const int x1add = wr ? LEAD + 1 : 3, x1msk = wr ? 127 : 7, x1shl = wr ? 7 : 9;
  const int x2add = wr ? LEAD : LEAD - 2, x2msk = wr ? 127 : 7, x2shl = wr ? 19 : 9;
  const unsigned x2base = wr ? sD0 : 0u;
  const unsigned stg = (unsigned)L_STG + (unsigned)wid * 1024u;
  const int stgl = (int)stg + lane * 16, chkl = (int)stg + (lane & 15) * 16;

#define SA(b, h) (((b) * 2 + (h)) * HTB)
#define SB(b, h) ((4 + (b) * 2 + (h)) * HTB)
#define STAGE_A(b, h, kt) asm volatile("s_mov_b32 m0, %0\n\ts_nop 0\n\tbuffer_load_dwordx4 %1, %2, %3 offen sc1 lds\n\tbuffer_load_dwordx4 %1, %2, %3 offen offset:1024 sc1 lds" \
    :: "s"(ldsw + SA(b, h)), "v"(voffA), "s"(srdA), "s"(sA0 + (unsigned)(kt) * KSTEP + (h) * HTB) : "m0", "memory")
#define STAGE_B(b, h, kt) asm volatile("s_mov_b32 m0, %0\n\ts_nop 0\n\tbuffer_load_dwordx4 %1, %3, %4 offen lds\n\tbuffer_load_dwordx4 %2, %3, %4 offen offset:1024 lds" \
    :: "s"(ldsw + SB(b, h)), "v"(voffB0), "v"(voffB1), "s"(srdB), "s"(sB0 + (unsigned)(kt) * KSTEP + (h) * 256) : "m0", "memory")
#define LDA(dst, b, h) do { _Pragma("unroll") for (int m = 0; m < 4; ++m) _Pragma("unroll") for (int k = 0; k < 2; ++k) dst[m][k] = *(const LAS bf16x8*)(lds + SA(b, h) + aoff + m * 2048 + k * 1024); } while (0)
#define TR(off) __builtin_amdgcn_ds_read_tr16_b64_v4i16((LAS s16x4*)(lds + (off)))
#define LDB(dst, b, h) do { _Pragma("unroll") for (int k = 0; k < 2; ++k) { \
    { s16x4 lo = TR(boff0 + ((b) * 2 + (h)) * HTB + k * 8192), hi = TR(boff0 + ((b) * 2 + (h)) * HTB + k * 8192 + 1024); dst[0][k] = __builtin_shufflevector(lo, hi, 0, 1, 2, 3, 4, 5, 6, 7); } \
    { s16x4 lo = TR(boff1 + ((b) * 2 + (h)) * HTB + k * 8192), hi = TR(boff1 + ((b) * 2 + (h)) * HTB + k * 8192 + 1024); dst[1][k] = __builtin_shufflevector(lo, hi, 0, 1, 2, 3, 4, 5, 6, 7); } } } while (0)
#define MMA(ai, bj, At, Bt) do { __builtin_amdgcn_s_setprio(1); _Pragma("unroll") for (int m = 0; m < 4; ++m) _Pragma("unroll") for (int n = 0; n < 2; ++n) _Pragma("unroll") for (int k = 0; k < 2; ++k) \
    acc[ai][bj][m][n] = __builtin_amdgcn_mfma_f32_16x16x32_bf16(Bt[n][k], At[m][k], acc[ai][bj][m][n], 0, 0, 0); __builtin_amdgcn_s_setprio(0); } while (0)
#define WAIT_V(n) asm volatile("s_waitcnt vmcnt(" #n ")" ::: "memory")
#define WAIT_L(n) asm volatile("s_waitcnt lgkmcnt(" #n ")" ::: "memory")
#define BAR __builtin_amdgcn_s_barrier()
#define SCHED __builtin_amdgcn_sched_barrier(0)
#define X1(t, sl) asm volatile("s_mov_b32 m0, %0\n\ts_nop 0\n\tbuffer_load_dwordx4 %1, %2, %3 offen sc0 sc1 lds" \
        :: "s"(ldsb + stg + (sl) * 8192), "v"(vx1), "s"(srdX1), "s"((unsigned)(((t) + x1add) & x1msk) << x1shl) : "m0", "memory")
#define X2_RD(sl) u32x4 pv_ = *(const LAS u32x4*)(lds + stgl + (sl) * 8192)
#define X2_ST(t) do { const int kk_ = (t) + x2add; pv_.x = wr ? pv_.x : base + (unsigned)kk_ + 1u; \
        __builtin_amdgcn_raw_buffer_store_b128(pv_, srdX2, vx2, kk_ < NKT ? x2base + ((unsigned)(kk_ & x2msk) << x2shl) : OOB, 17); asm volatile("" ::: "memory"); } while (0)
#define CHECK(kp, sl) do { if (!wr && (kp) < NKT) { const unsigned fv_ = *(const LAS unsigned*)(lds + chkl + (sl) * 8192); \
        if (!__all((int)(fv_ - (base + (unsigned)(kp) + 1u)) >= 0)) spin_flags(myflags + ((kp) & 7) * 128, base + (unsigned)(kp) + 1u, tmo, call, dead); } } while (0)

  if (tid == 0) *dead = 0u;
  {
    const int k = (wid + 4) & 7;
    if (k < LEAD - 2) {
      int R, C; stage_rc((unit & 7) * 2048 + lane * 16, R, C);
      const unsigned s0 = (unsigned)(((unit >> 3) * 128 + R) * KT + C) * 2u;
      const u32x4 v0 = __builtin_amdgcn_raw_buffer_load_b128(srdX, s0, (unsigned)k * 128u, 0);
      const u32x4 v1 = __builtin_amdgcn_raw_buffer_load_b128(srdX, s0 + 64u, (unsigned)k * 128u, 0);
      while (__builtin_amdgcn_s_memrealtime() - t_start < (unsigned long long)(k * PACE)) __builtin_amdgcn_s_sleep(1);
      __builtin_amdgcn_raw_buffer_store_b128(v0, srdD, (unsigned)lane * 16u, sD0 + (unsigned)k * KSTEP, 17);
      __builtin_amdgcn_raw_buffer_store_b128(v1, srdD, (unsigned)lane * 16u + 1024u, sD0 + (unsigned)k * KSTEP, 17);
      WAIT_V(0);
      __builtin_amdgcn_raw_buffer_store_b32(base + (unsigned)k + 1u, srdFd, lane == 0 ? (unsigned)(rank * 4096 + unit * 16) : OOB, (unsigned)k * 512u, 17);
      asm volatile("" ::: "memory");
    }
  }
  STAGE_B(0, 0, 0); STAGE_B(0, 1, 0); STAGE_B(1, 0, 1); STAGE_B(1, 1, 1);
  if (wr) {
    X1(-3, 0); X1(-2, 1);
    WAIT_V(0);
    while (__builtin_amdgcn_s_memrealtime() - t_start < (unsigned long long)((LEAD - 2) * PACE)) __builtin_amdgcn_s_sleep(1);
    { X2_RD(0); X2_ST(-2); } { X2_RD(1); X2_ST(-1); }
    WAIT_L(0);
    X1(-1, 1);
    WAIT_V(2);
  } else {
    WAIT_L(0);
    spin_flags(myflags, base + 1u, tmo, call, dead);
  }
  BAR;
  STAGE_A(0, 0, 0); STAGE_A(0, 1, 0);
  if (!wr) { spin_flags(myflags + 128, base + 2u, tmo, call, dead); spin_flags(myflags + 256, base + 3u, tmo, call, dead); }
  BAR;
  STAGE_A(1, 0, 1);
  f32x4 acc[2][2][4][2];
#pragma unroll
  for (int i = 0; i < 2; ++i)
#pragma unroll
    for (int j = 0; j < 2; ++j)
#pragma unroll
      for (int m = 0; m < 4; ++m)
#pragma unroll
        for (int n = 0; n < 2; ++n) acc[i][j][m][n] = (f32x4){0.f, 0.f, 0.f, 0.f};
  bf16x8 At[4][2], B0[2][2], B1[2][2];
  WAIT_V(2); BAR;
  if (wr == 1) BAR;
  for (int t = 0; t < NKT; t += 2) {
    const int t1 = t + 1, t2 = t + 2, t3 = t + 3;
    LDB(B0, 0, 0); SCHED; LDA(At, 0, 0); X1(t, 0); STAGE_A(1, 1, t1);
    WAIT_L(8); BAR; WAIT_L(0); MMA(0, 0, At, B0); BAR; SCHED;
    { X2_RD(1); SCHED; LDB(B1, 0, 1); X2_ST(t); } STAGE_B(0, 0, t2);
    BAR; WAIT_L(0); MMA(0, 1, At, B1); BAR;
    LDA(At, 0, 1); STAGE_A(0, 0, t2);
    BAR; WAIT_L(0); MMA(1, 0, At, B0); BAR; SCHED;
    STAGE_B(0, 1, t2);
    WAIT_V(7); CHECK(t + 3, 0); BAR; MMA(1, 1, At, B1); BAR;
    LDB(B0, 1, 0); SCHED; LDA(At, 1, 0); X1(t1, 1); STAGE_A(0, 1, t2);
    WAIT_L(8); BAR; WAIT_L(0); MMA(0, 0, At, B0); BAR; SCHED;
    { X2_RD(0); SCHED; LDB(B1, 1, 1); X2_ST(t1); } STAGE_B(1, 0, t3);
    BAR; WAIT_L(0); MMA(0, 1, At, B1); BAR;
    LDA(At, 1, 1); STAGE_A(1, 0, t3);
    BAR; WAIT_L(0); MMA(1, 0, At, B0); BAR; SCHED;
    STAGE_B(1, 1, t3);
    WAIT_V(7); CHECK(t + 4, 1); BAR; MMA(1, 1, At, B1); BAR;
  }
  if (wr == 0) BAR;
  {
    int tz = threadIdx.x; asm volatile("" : "+v"(tz));
    const int lane2 = tz & 63, fr2 = lane2 & 15, fq2 = lane2 >> 4;
    const bool odd = fq2 & 1;
    const bool bad = __builtin_amdgcn_readfirstlane(*dead) != 0u;
    char* obase = a.out + ((size_t)(pm * BM + wr * 64 + fr2) * NL + (size_t)(pn * BM + wc * 32 + 8 * fq2)) * 2;
#pragma unroll
    for (int ai = 0; ai < 2; ++ai)
#pragma unroll
      for (int m = 0; m < 4; ++m)
#pragma unroll
        for (int bj = 0; bj < 2; ++bj) {
          const f32x4 v0 = acc[ai][bj][m][0], v1 = acc[ai][bj][m][1];
          const unsigned a0 = cvt_pk_bf16(gelu1(v0[0]), gelu1(v0[1])), a1 = cvt_pk_bf16(gelu1(v0[2]), gelu1(v0[3]));
          const unsigned b0 = cvt_pk_bf16(gelu1(v1[0]), gelu1(v1[1])), b1 = cvt_pk_bf16(gelu1(v1[2]), gelu1(v1[3]));
          u32x4 w; w.x = odd ? b0 : a0; w.y = odd ? b1 : a1; w.z = odd ? a0 : b0; w.w = odd ? a1 : b1;
          if (bad) w = (u32x4){0x7fc07fc0u, 0x7fc07fc0u, 0x7fc07fc0u, 0x7fc07fc0u};
          *(u32x4*)(obase + (size_t)(ai * HALF + m * 16) * (NL * 2) + bj * (HALF * 2)) = w;
        }
    if (tz == 0) __hip_atomic_store((unsigned*)(ws + WS_CTR + (size_t)bid * 128), call, __ATOMIC_RELAXED, __HIP_MEMORY_SCOPE_AGENT);
  }
  WAIT_V(0);
}
}

extern "C" void kernel_launch(int rank, int nranks, ncclComm_t comm,
                              hipStream_t stream,
                              void* const* d_in, const int* in_sizes, int n_in,
                              void* d_out, int out_size,
                              const int* in_dtypes, int out_dtype,
                              void* const* d_out_all,
                              void* const* d_ws_all, size_t ws_bytes) {
  using namespace agg;
  if (nranks != NR || (unsigned)rank >= (unsigned)NR || n_in < 2 || ws_bytes < WS_NEED || in_sizes[0] != MS * KT || in_sizes[1] != KT * NL || out_size != MT * NL) {
    fprintf(stderr, "kernel_launch: unexpected problem shape\n");
    return;
  }
  static unsigned attr_done = 0;
  int dev = 0; HIP_CHECK(hipGetDevice(&dev));
  if (!(attr_done >> (dev & 31) & 1u)) {
    HIP_CHECK(hipFuncSetAttribute((const void*)ag_gemm_kernel, hipFuncAttributeMaxDynamicSharedMemorySize, LDS_BYTES));
    attr_done |= 1u << (dev & 31);
  }
  Args a;
  a.x = (const char*)d_in[0]; a.W = (const char*)d_in[1]; a.out = (char*)d_out; a.rank = rank;
  for (int p = 0; p < NR; ++p) a.ws[p] = (char*)d_ws_all[p];
  hipLaunchKernelGGL(ag_gemm_kernel, dim3(256), dim3(512), LDS_BYTES, stream, a);
  HIP_CHECK(hipGetLastError());
}
```

```cpp
#include <hip/hip_runtime.h>
#include <rccl/rccl.h>
#include <cstdio>

#define HIP_CHECK(expr) do { hipError_t _e = (expr); if (_e != hipSuccess) { \
    fprintf(stderr, "HIP error %s at %s:%d\n", hipGetErrorString(_e), __FILE__, __LINE__); \
    return; } } while (0)

namespace agg {
#define LAS __attribute__((address_space(3)))
typedef short bf16x8 __attribute__((ext_vector_type(8)));
typedef short s16x4 __attribute__((ext_vector_type(4)));
typedef float f32x4 __attribute__((ext_vector_type(4)));
typedef float f32x2 __attribute__((ext_vector_type(2)));
typedef unsigned u32x4 __attribute__((ext_vector_type(4)));

constexpr int NR = 8;
constexpr int MT = 4096, KT = 4096, NL = 4096;
constexpr int MS = MT / NR;
constexpr int BM = 256, BK = 64, HALF = 128, HTB = 16384, NKT = KT / BK;
constexpr unsigned KSTEP = 32u * HTB;
static_assert(KSTEP == 64u * NL * 2u, "");
constexpr unsigned IMG_BYTES = (unsigned)MT * KT * 2;
constexpr unsigned X_BYTES = (unsigned)MS * KT * 2, W_BYTES = (unsigned)KT * NL * 2;
constexpr size_t WS_CTR = 0;
constexpr size_t WS_TMO = 32768;
constexpr size_t WS_FLAG = 65536;
constexpr unsigned FLAG_BYTES = NR * 8 * 128;
constexpr size_t WS_STAMP = 262144;
constexpr size_t WS_IMG = 1u << 20;
constexpr size_t WS_NEED = WS_IMG + 2 * (size_t)IMG_BYTES;
constexpr int L_STG = 8 * HTB;
constexpr int L_DEAD = L_STG + 8192;
constexpr int L_PRO = L_STG + 16384;
constexpr int LDS_BYTES = L_PRO + 16384;
static_assert(LDS_BYTES <= 160 * 1024, "");
constexpr unsigned OOB = 0x40000000u;
#define LEAD 6
#define EAGER 0
#define NPRO (EAGER ? LEAD - 1 : LEAD - 2)
static_assert(NPRO <= 4, "");
#define BDELAY 40
#define PACE 110
constexpr unsigned long long DEADLINE = 50000000ull;

struct Args { const char* x; const char* W; char* out; char* ws[NR]; int rank; };

__host__ __device__ __forceinline__ int lds_byte(int r, int c) { const int st = (r >> 4) * 2 + (c >> 5), rr = r & 15, cc = c & 31, ob = rr * 64 + cc * 2; return st * 1024 + (ob ^ (((ob >> 9) & 1) << 5)); }
__host__ __device__ __forceinline__ void stage_rc(int b, int& R, int& C) { const int st = b / 1024, sb = b % 1024, swz = sb ^ (((sb >> 9) & 1) << 5); R = (st >> 1) * 16 + swz / 64; C = (st & 1) * 32 + (swz % 64) / 2; }
__host__ __device__ __forceinline__ int fk(int k) { return (k & 1) | ((k & 2) << 1) | (k & 8); }

__device__ __forceinline__ unsigned cvt_pk_bf16(float lo, float hi) { unsigned r; asm("v_cvt_pk_bf16_f32 %0, %1, %2" : "=v"(r) : "v"(lo), "v"(hi)); return r; }
__device__ __forceinline__ unsigned gelu2(f32x2 y) {
  const float k1 = -2.0f * 0.7978845608028654f * 1.4426950408889634f, k2 = k1 * 0.044715f;
  const f32x2 z = y * __builtin_elementwise_fma(y * y, (f32x2){k2, k2}, (f32x2){k1, k1});
  f32x2 e; e.x = __builtin_amdgcn_exp2f(z.x); e.y = __builtin_amdgcn_exp2f(z.y);
  const f32x2 d = e + 1.0f;
  f32x2 r; r.x = __builtin_amdgcn_rcpf(d.x); r.y = __builtin_amdgcn_rcpf(d.y);
  const f32x2 o = y * r;
  return cvt_pk_bf16(o.x, o.y);
}
template <int AUX> __device__ __forceinline__ void st128(u32x4 v, __amdgpu_buffer_rsrc_t r, unsigned voff, unsigned soff) {
  __builtin_amdgcn_raw_buffer_store_b128(v, r, voff, soff, AUX);
  asm volatile("s_nop 1" :: "v"(v) : "memory");
}
__device__ __forceinline__ __amdgpu_buffer_rsrc_t mk_rsrc(const void* p, unsigned bytes) { return __builtin_amdgcn_make_buffer_rsrc((void*)p, (short)0, (int)bytes, 0x00020000); }

__device__ __forceinline__ void spin_flags(const unsigned* f, unsigned need, unsigned* tmo, unsigned call, LAS unsigned* dead) {
  const unsigned long long t0 = __builtin_amdgcn_s_memrealtime();
  __builtin_amdgcn_s_waitcnt(0xC07F);
  for (;;) {
    const unsigned v = __hip_atomic_load(f, __ATOMIC_RELAXED, __HIP_MEMORY_SCOPE_SYSTEM);
    if (__all((int)(v - need) >= 0)) break;
    if (__builtin_amdgcn_readfirstlane(__hip_atomic_load(tmo, __ATOMIC_RELAXED, __HIP_MEMORY_SCOPE_AGENT)) == call ||
        __builtin_amdgcn_s_memrealtime() - t0 > DEADLINE) {
      __hip_atomic_store(tmo, call, __ATOMIC_RELAXED, __HIP_MEMORY_SCOPE_AGENT);
      *dead = 1u;
      asm volatile("s_waitcnt vmcnt(0) lgkmcnt(0)" ::: "memory");
      break;
    }
    __builtin_amdgcn_s_sleep(1);
  }
  asm volatile("" ::: "memory");
}

__global__ __launch_bounds__(512, 2) void ag_gemm_kernel(const Args a) {
  extern __shared__ __attribute__((aligned(16))) unsigned char shm_[];
  LAS unsigned char* lds = (LAS unsigned char*)shm_;
  const int tid = threadIdx.x, wid = __builtin_amdgcn_readfirstlane(tid >> 6), lane = tid & 63, wr = wid >> 2, wc = wid & 3, fr = lane & 15, fq = lane >> 4;
  const int bid = blockIdx.x, rank = a.rank;
  char* const ws = a.ws[0];
  unsigned prev;
  asm volatile("s_load_dword %0, %1, 0x0" : "=s"(prev) : "s"(ws + WS_CTR + (size_t)bid * 128) : "memory");
  const int xcd = bid & 7, idx = bid >> 3;
  const int pm = (xcd & 1) * 8 + (idx & 7), pn = (xcd >> 1) * 4 + (idx >> 3);
  const int unit = (bid & 7) | ((bid >> 6) << 3);
  char* const wsd = a.ws[(bid >> 3) & (NR - 1)];
  const __amdgpu_buffer_rsrc_t srdB = mk_rsrc(a.W, W_BYTES);
  const __amdgpu_buffer_rsrc_t srdX = mk_rsrc(a.x, X_BYTES);
  const __amdgpu_buffer_rsrc_t srdFd = mk_rsrc(wsd + WS_FLAG, FLAG_BYTES);
  const __amdgpu_buffer_rsrc_t srdFl = mk_rsrc(ws + WS_FLAG, FLAG_BYTES);
  const unsigned ldsb = (unsigned)(size_t)lds;
  const unsigned stg = (unsigned)L_STG + (unsigned)wid * 1024u;

  unsigned vx1, vx2;
  {
    int R, C; stage_rc((unit & 7) * 2048 + wc * 512 + (lane & 31) * 16, R, C);
    const unsigned src = (unsigned)(((unit >> 3) * 128 + R) * KT + C) * 2u;
    const unsigned dst = (unsigned)(wc * 512 + (lane & 31) * 16);
    if (wr) { vx1 = lane < 32 ? src : OOB; vx2 = lane < 32 ? dst : OOB; }
    else { vx1 = lane < 4 ? (unsigned)((pm >> 1) * 1024 + (pm & 1) * 64 + lane * 16) : OOB; vx2 = (tid == 0) ? (unsigned)(rank * 1024 + unit * 4) : OOB; }
  }
#define XLOAD(kx, sl) asm volatile("s_mov_b32 m0, %0\n\ts_nop 0\n\tbuffer_load_dwordx4 %1, %2, %3 offen lds" \
        :: "s"(ldsb + stg + (sl) * 8192), "v"(vx1), "s"(srdX), "s"((unsigned)(kx) << 7) : "m0", "memory")
#define POLL(kp) asm volatile("s_mov_b32 m0, %0\n\ts_nop 0\n\tbuffer_load_dwordx4 %1, %2, %3 offen sc0 sc1 lds" \
        :: "s"(ldsb + stg), "v"(vx1), "s"(srdFl), "s"((unsigned)((kp) & 7) << 7) : "m0", "memory")

#define PLOAD(voff, kx, i) asm volatile("s_mov_b32 m0, %0\n\ts_nop 0\n\tbuffer_load_dwordx4 %1, %2, %3 offen lds" \
        :: "s"(ldsb + L_PRO + wc * 4096 + (i) * 1024), "v"(voff), "s"(srdX), "s"((unsigned)(kx) << 7) : "m0", "memory")
#define PREAD(i) (*(const LAS u32x4*)(lds + L_PRO + wc * 4096 + (i) * 1024 + lane * 16))
  if (wr) {
    int R, C; stage_rc((unit & 7) * 2048 + lane * 16, R, C);
    const unsigned s0 = (unsigned)(((unit >> 3) * 128 + R) * KT + C) * 2u;
    PLOAD(s0, wc, 0); PLOAD(s0 + 64u, wc, 1);
    PLOAD(vx1, LEAD - 2, 3); PLOAD(vx1, LEAD - 1, 2);
    XLOAD(LEAD, 0); XLOAD(LEAD + 1, 1);
  }
  asm volatile("s_waitcnt lgkmcnt(0)" : "+s"(prev) :: "memory");
  const unsigned call = prev + 1u, base = prev * (unsigned)NKT, par = call & 1u;
  const unsigned long long t_start = __builtin_amdgcn_s_memrealtime();
  const __amdgpu_buffer_rsrc_t srdA = mk_rsrc(ws + WS_IMG + (size_t)par * IMG_BYTES, IMG_BYTES);
  const __amdgpu_buffer_rsrc_t srdD = mk_rsrc(wsd + WS_IMG + (size_t)par * IMG_BYTES, IMG_BYTES);
  unsigned* const tmo = (unsigned*)(ws + WS_TMO);
  const unsigned* const myflags = (const unsigned*)(ws + WS_FLAG) + (pm >> 1) * 256 + (pm & 1) * 16 + (lane & 15);
  LAS unsigned* const dead = (LAS unsigned*)(lds + L_DEAD);

  const unsigned voffA = (unsigned)(wid * 2048 + lane * 16);
  const int kB = wid * 8 + (lane >> 4);
  const unsigned voffB0 = (unsigned)kB * (NL * 2) + (unsigned)(((lane & 15) ^ fk(kB)) * 16);
  const unsigned voffB1 = voffB0 + 4 * (NL * 2) - 1024;
  const unsigned ldsw = ldsb + (unsigned)wid * 2048u;
  const int aoff = lds_byte(wr * 64 + fr, fq * 8);
  const int q_ = (lane & 15) >> 2, p_ = lane & 3;
  const int f_ = (q_ & 1) | ((q_ & 2) << 1) | ((fq & 1) << 3);
  int boff0 = 4 * HTB + 256 * (8 * fq + q_) + 16 * ((4 * wc + p_) ^ f_) + 8 * (p_ & 1);
  int boff1 = boff0 ^ 8;
  asm volatile("" : "+v"(boff0), "+v"(boff1));
  const unsigned sA0 = (unsigned)(2 * pm) * HTB, sB0 = (unsigned)pn * 512u;
  const unsigned sD0 = (unsigned)(rank * 4 + (unit >> 3)) * HTB + (unsigned)(unit & 7) * 2048u;
  const int stgl = (int)stg + lane * 16, chkl = (int)stg + (lane & 15) * 4;

#define SA(b, h) (((b) * 2 + (h)) * HTB)
#define SB(b, h) ((4 + (b) * 2 + (h)) * HTB)
#define STAGE_A_(b, h, kt, w4) asm volatile("s_mov_b32 m0, %0\n\ts_nop 0\n\tbuffer_load_dwordx4 %1, %2, %3 offen sc1 lds\n\tbuffer_load_dwordx4 %1, %2, %3 offen offset:1024 sc1 lds" \
    :: "s"(ldsw + SA(b, h) + (w4) * 8192), "v"(voffA), "s"(srdA), "s"(sA0 + (unsigned)(kt) * KSTEP + (h) * HTB + (w4) * 8192) : "m0", "memory")
#define STAGE_B_(b, h, kt, w4) asm volatile("s_mov_b32 m0, %0\n\ts_nop 0\n\tbuffer_load_dwordx4 %1, %3, %4 offen lds\n\tbuffer_load_dwordx4 %2, %3, %4 offen offset:1024 lds" \
    :: "s"(ldsw + SB(b, h) + (w4) * 8192), "v"(voffB0), "v"(voffB1), "s"(srdB), "s"(sB0 + (unsigned)(kt) * KSTEP + (h) * 256 + (w4) * 32 * (NL * 2)) : "m0", "memory")
#define PA_(b, h, kt, i) asm volatile("s_mov_b32 m0, %0\n\ts_nop 0\n\tbuffer_load_dwordx4 %1, %2, %3 offen offset:" #i " sc1 lds" \
    :: "s"(ldsw + SA(b, h)), "v"(voffA), "s"(srdA), "s"(sA0 + (unsigned)(kt) * KSTEP + (h) * HTB) : "m0", "memory")
#define PB_(b, h, kt, v, i) asm volatile("s_mov_b32 m0, %0\n\ts_nop 0\n\tbuffer_load_dwordx4 %1, %2, %3 offen offset:" #i " lds" \
    :: "s"(ldsw + SB(b, h)), "v"(v), "s"(srdB), "s"(sB0 + (unsigned)(kt) * KSTEP + (h) * 256) : "m0", "memory")
#define LDA2(dst, b, h, m0_) do { _Pragma("unroll") for (int m = m0_; m < m0_ + 2; ++m) _Pragma("unroll") for (int k = 0; k < 2; ++k) dst[m][k] = *(const LAS bf16x8*)(lds + SA(b, h) + aoff + m * 2048 + k * 1024); } while (0)
#define LDB1(dst, b, h, k) do { \
    { s16x4 lo = TR(boff0 + ((b) * 2 + (h)) * HTB + k * 8192), hi = TR(boff0 + ((b) * 2 + (h)) * HTB + k * 8192 + 1024); dst[0][k] = __builtin_shufflevector(lo, hi, 0, 1, 2, 3, 4, 5, 6, 7); } \
    { s16x4 lo = TR(boff1 + ((b) * 2 + (h)) * HTB + k * 8192), hi = TR(boff1 + ((b) * 2 + (h)) * HTB + k * 8192 + 1024); dst[1][k] = __builtin_shufflevector(lo, hi, 0, 1, 2, 3, 4, 5, 6, 7); } } while (0)
#define LDA(dst, b, h) do { _Pragma("unroll") for (int m = 0; m < 4; ++m) _Pragma("unroll") for (int k = 0; k < 2; ++k) dst[m][k] = *(const LAS bf16x8*)(lds + SA(b, h) + aoff + m * 2048 + k * 1024); } while (0)
#define TR(off) __builtin_amdgcn_ds_read_tr16_b64_v4i16((LAS s16x4*)(lds + (off)))
#define LDB(dst, b, h) do { _Pragma("unroll") for (int k = 0; k < 2; ++k) { \
    { s16x4 lo = TR(boff0 + ((b) * 2 + (h)) * HTB + k * 8192), hi = TR(boff0 + ((b) * 2 + (h)) * HTB + k * 8192 + 1024); dst[0][k] = __builtin_shufflevector(lo, hi, 0, 1, 2, 3, 4, 5, 6, 7); } \
    { s16x4 lo = TR(boff1 + ((b) * 2 + (h)) * HTB + k * 8192), hi = TR(boff1 + ((b) * 2 + (h)) * HTB + k * 8192 + 1024); dst[1][k] = __builtin_shufflevector(lo, hi, 0, 1, 2, 3, 4, 5, 6, 7); } } } while (0)
#define WAIT_V(n) asm volatile("s_waitcnt vmcnt(" #n ")" ::: "memory")
#define WAIT_L(n) do { asm volatile("" ::: "memory"); __builtin_amdgcn_s_waitcnt(0xC07F | ((n) << 8)); asm volatile("" ::: "memory"); } while (0)
#define WAIT_V0 do { asm volatile("" ::: "memory"); __builtin_amdgcn_s_waitcnt(0x0F70); asm volatile("" ::: "memory"); } while (0)
#define BAR __builtin_amdgcn_s_barrier()
#define SCHED __builtin_amdgcn_sched_barrier(0)
#define MMA(ai, bj, At, Bt) do { _Pragma("unroll") for (int m = 0; m < 4; ++m) _Pragma("unroll") for (int n = 0; n < 2; ++n) _Pragma("unroll") for (int k = 0; k < 2; ++k) { \
    acc[ai][bj][m][n] = __builtin_amdgcn_mfma_f32_16x16x32_bf16(Bt[n][k], At[m][k], acc[ai][bj][m][n], 0, 0, 0); SCHED; } } while (0)
#define MMA2(ai, At, B0, B1) do { SCHED; MMA(ai, 0, At, B0); MMA(ai, 1, At, B1); SCHED; } while (0)
#define STAMP(i) do {} while (0)
#define STAMPC(i) do {} while (0)
#define WAIT_VT(n) WAIT_V(n)
#define TIMED(acc_, stmt) stmt
#define SLOWCNT do {} while (0)
#define PUSH_RD(sl) (*(const LAS u32x4*)(lds + stgl + (sl) * 8192))
#define PUSH_ST(v, t) st128<17>(v, srdD, vx2, sD0 + (unsigned)((t) + LEAD) * KSTEP)
#define PUBLISH(kf) do { __builtin_amdgcn_raw_buffer_store_b32(base + (unsigned)(kf) + 1u, srdFd, vx2, (kf) < NKT ? (unsigned)((kf) & 7) * 128u : OOB, 17); asm volatile("" ::: "memory"); } while (0)
#define CHECK_ON true
#define CHECK(fv, kp) do { if (__builtin_expect(CHECK_ON && !wr && (kp) < NKT && !__all((int)((fv) - (base + (unsigned)(kp) + 1u)) >= 0), 0)) { \
        SLOWCNT; spin_flags(myflags + ((kp) & 7) * 32, base + (unsigned)(kp) + 1u, tmo, call, dead); } } while (0)
#define PACE_UNTIL(ticks) do { while (__builtin_amdgcn_s_memrealtime() - t_start < (unsigned long long)(ticks)) __builtin_amdgcn_s_sleep(1); } while (0)

  if (wr) {
    if (wc < NPRO) {
      WAIT_V(4);
      const u32x4 pv0 = PREAD(0), pv1 = PREAD(1);
      PACE_UNTIL(wc * PACE);
      st128<17>(pv0, srdD, (unsigned)lane * 16u, sD0 + (unsigned)wc * KSTEP);
      st128<17>(pv1, srdD, (unsigned)lane * 16u + 1024u, sD0 + (unsigned)wc * KSTEP);
      WAIT_V(0);
      __builtin_amdgcn_raw_buffer_store_b32(base + (unsigned)wc + 1u, srdFd, lane == 0 ? (unsigned)(rank * 1024 + unit * 4) : OOB, (unsigned)wc * 128u, 17);
      asm volatile("" ::: "memory");
    } else WAIT_V(0);
    STAMP(1);
    const u32x4 pv2 = PREAD(2), pv3 = PREAD(3);
    PACE_UNTIL(NPRO * PACE);
    PUSH_ST(pv3, -2);
    PACE_UNTIL((NPRO + 1) * PACE);
    PUSH_ST(pv2, -1);
    WAIT_V(1);
    STAMP(2);
  } else {
    if (tid == 0) *dead = 0u;
    PACE_UNTIL(BDELAY);
    STAGE_B_(0, 0, 0, 0); STAGE_B_(0, 0, 0, 1); STAGE_B_(0, 1, 0, 0); STAGE_B_(0, 1, 0, 1);
    STAGE_B_(1, 0, 1, 0); STAGE_B_(1, 0, 1, 1); STAGE_B_(1, 1, 1, 0); STAGE_B_(1, 1, 1, 1);
    STAMP(1);
    spin_flags(myflags, base + 1u, tmo, call, dead);
    STAMP(2);
    STAGE_A_(0, 0, 0, 0); STAGE_A_(0, 0, 0, 1); STAGE_A_(0, 1, 0, 0); STAGE_A_(0, 1, 0, 1);
    spin_flags(myflags + 32, base + 2u, tmo, call, dead);
    STAMP(3);
    WAIT_V(0);
    STAGE_A_(1, 0, 1, 0); STAGE_A_(1, 0, 1, 1);
  }
  f32x4 acc[2][2][4][2];
#pragma unroll
  for (int i = 0; i < 2; ++i)
#pragma unroll
    for (int j = 0; j < 2; ++j)
#pragma unroll
      for (int m = 0; m < 4; ++m)
#pragma unroll
        for (int n = 0; n < 2; ++n) acc[i][j][m][n] = (f32x4){0.f, 0.f, 0.f, 0.f};
  bf16x8 At[4][2], B0a[2][2], B0b[2][2], B1[2][2];
  WAIT_L(0);
  STAMP(4);
  BAR;
  LDB(B0a, 0, 0);
  WAIT_L(0);
  BAR;
  STAMP(5); STAMPC(44);
  if (wr == 1) BAR;
#define LA_X(t) do { if (wr) PUSH_ST(pv_, t); else PUBLISH((t) + LEAD - 2); } while (0)
#define LB_W WAIT_V(5)
#define LB_X(t, sl) do { if (wr) XLOAD((t) + LEAD + 2, sl); } while (0)
#define W_A WAIT_VT(11)
#define KTILE(t, b, B0c, B0n) do { \
    { if (!wr) POLL((t) + 2); const u32x4 pv_ = PUSH_RD(b); \
      PA_((b) ^ 1, 1, (t) + 1, 0); LDB1(B1, b, 1, 0); PA_((b) ^ 1, 1, (t) + 1, 1024); LDB1(B1, b, 1, 1); \
      PB_(b, 0, (t) + 2, voffB0, 0); LDA2(At, b, 0, 0); PB_(b, 0, (t) + 2, voffB1, 1024); LDA2(At, b, 0, 2); LA_X(t); } \
    W_A; WAIT_L(0); BAR; MMA2(0, At, B0c, B1); BAR; \
    { TIMED(wsum3, LB_W); const unsigned fv_ = *(const LAS unsigned*)(lds + chkl); LB_X(t, b); \
      PB_(b, 1, (t) + 2, voffB0, 0); LDA2(At, b, 1, 0); PB_(b, 1, (t) + 2, voffB1, 1024); LDA2(At, b, 1, 2); \
      CHECK(fv_, (t) + 2); } \
    PA_(b, 0, (t) + 2, 0); LDB1(B0n, (b) ^ 1, 0, 0); PA_(b, 0, (t) + 2, 1024); LDB1(B0n, (b) ^ 1, 0, 1); \
    WAIT_L(0); BAR; MMA2(1, At, B0c, B1); BAR; } while (0)
  for (int t = 0; t < NKT; t += 2) {
    KTILE(t, 0, B0a, B0b);
    KTILE(t + 1, 1, B0b, B0a);
  }
  STAMPC(45);
  if (wr == 0) BAR;
  {
    int tz = threadIdx.x; asm volatile("" : "+v"(tz));
    const int lane2 = tz & 63, fr2 = lane2 & 15, fq2 = lane2 >> 4;
    const bool odd = fq2 & 1;
    const bool bad = __builtin_amdgcn_readfirstlane(*dead) != 0u;
    const __amdgpu_buffer_rsrc_t srdO = mk_rsrc(a.out, (unsigned)MT * NL * 2);
    const unsigned ooff = ((unsigned)(pm * BM + wr * 64 + fr2) * NL + (unsigned)(pn * BM + wc * 32 + 8 * fq2)) * 2u;
    if (bad) {
#pragma unroll
      for (int i = 0; i < 2; ++i)
#pragma unroll
        for (int j = 0; j < 2; ++j)
#pragma unroll
          for (int m = 0; m < 4; ++m)
#pragma unroll
            for (int n = 0; n < 2; ++n) acc[i][j][m][n] = (f32x4){__builtin_nanf(""), __builtin_nanf(""), __builtin_nanf(""), __builtin_nanf("")};
    }
#pragma unroll
    for (int ai = 0; ai < 2; ++ai)
#pragma unroll
      for (int m = 0; m < 4; ++m)
#pragma unroll
        for (int bj = 0; bj < 2; ++bj) {
          const f32x4 v0 = acc[ai][bj][m][0], v1 = acc[ai][bj][m][1];
          const unsigned a0 = gelu2((f32x2){v0[0], v0[1]}), a1 = gelu2((f32x2){v0[2], v0[3]});
          const unsigned b0 = gelu2((f32x2){v1[0], v1[1]}), b1 = gelu2((f32x2){v1[2], v1[3]});
          u32x4 w; w.x = odd ? b0 : a0; w.y = odd ? b1 : a1; w.z = odd ? a0 : b0; w.w = odd ? a1 : b1;
          st128<16>(w, srdO, ooff + bj * (HALF * 2), (unsigned)(ai * HALF + m * 16) * (NL * 2));
        }
    STAMP(40);
    if (tz == 0) __hip_atomic_store((unsigned*)(ws + WS_CTR + (size_t)bid * 128), call, __ATOMIC_RELAXED, __HIP_MEMORY_SCOPE_AGENT);
  }
}
}

extern "C" void kernel_launch(int rank, int nranks, ncclComm_t comm,
                              hipStream_t stream,
                              void* const* d_in, const int* in_sizes, int n_in,
                              void* d_out, int out_size,
                              const int* in_dtypes, int out_dtype,
                              void* const* d_out_all,
                              void* const* d_ws_all, size_t ws_bytes) {
  using namespace agg;
  if (nranks != NR || (unsigned)rank >= (unsigned)NR || n_in < 2 || ws_bytes < WS_NEED || in_sizes[0] != MS * KT || in_sizes[1] != KT * NL || out_size != MT * NL) {
    fprintf(stderr, "kernel_launch: unexpected problem shape\n");
    return;
  }
  static unsigned attr_done = 0;
  int dev = 0; HIP_CHECK(hipGetDevice(&dev));
  if (!(attr_done >> (dev & 31) & 1u)) {
    HIP_CHECK(hipFuncSetAttribute((const void*)ag_gemm_kernel, hipFuncAttributeMaxDynamicSharedMemorySize, LDS_BYTES));
    attr_done |= 1u << (dev & 31);
  }
  Args a;
  a.x = (const char*)d_in[0]; a.W = (const char*)d_in[1]; a.out = (char*)d_out; a.rank = rank;
  for (int p = 0; p < NR; ++p) a.ws[p] = (char*)d_ws_all[(rank + p) & (NR - 1)];
  hipLaunchKernelGGL(ag_gemm_kernel, dim3(256), dim3(512), LDS_BYTES, stream, a);
  HIP_CHECK(hipGetLastError());
}
```

```cpp
#include <hip/hip_runtime.h>
#include <rccl/rccl.h>
#include <cstdio>

#define HIP_CHECK(expr) do { hipError_t _e = (expr); if (_e != hipSuccess) { \
    fprintf(stderr, "HIP error %s at %s:%d\n", hipGetErrorString(_e), __FILE__, __LINE__); \
    return; } } while (0)

namespace agg {
#define LAS __attribute__((address_space(3)))
typedef short bf16x8 __attribute__((ext_vector_type(8)));
typedef short s16x4 __attribute__((ext_vector_type(4)));
typedef float f32x4 __attribute__((ext_vector_type(4)));
typedef float f32x2 __attribute__((ext_vector_type(2)));
typedef unsigned u32x4 __attribute__((ext_vector_type(4)));

constexpr int NR = 8;
constexpr int MT = 4096, KT = 4096, NL = 4096;
constexpr int MS = MT / NR;
constexpr int BM = 256, BK = 64, HALF = 128, HTB = 16384, NKT = KT / BK;
constexpr unsigned KSTEP = 32u * HTB;
static_assert(KSTEP == 64u * NL * 2u, "");
constexpr unsigned IMG_BYTES = (unsigned)MT * KT * 2;
constexpr unsigned X_BYTES = (unsigned)MS * KT * 2, W_BYTES = (unsigned)KT * NL * 2;
constexpr size_t WS_CTR = 0;
constexpr size_t WS_TMO = 32768;
#define FSTR 4
constexpr size_t WS_FLAG = 65536;
constexpr unsigned FSLOT = 32 * FSTR, FSRC = 8 * FSLOT, FLAG_BYTES = NR * FSRC;
static_assert((FSTR == 4 || FSTR >= 16) && WS_FLAG + FLAG_BYTES <= 524288, "");
#define POLL_SYS 0
#define POLL_SC " sc1"
#define POLL_SCOPE __HIP_MEMORY_SCOPE_AGENT
constexpr size_t WS_STAMP = 524288;
constexpr size_t WS_IMG = 1u << 20;
constexpr size_t WS_NEED = WS_IMG + 2 * (size_t)IMG_BYTES;
constexpr int L_STG = 8 * HTB;
constexpr int L_DEAD = L_STG + 8192 + 4096;
constexpr int L_PRO = L_STG + 16384;
constexpr int LDS_BYTES = L_PRO + 16384;
static_assert(LDS_BYTES <= 160 * 1024, "");
constexpr unsigned OOB = 0x40000000u;
#define LEAD 5
#define NPRO (LEAD - 1)
static_assert(NPRO <= 8, "");
#define PDEL 1
#define DPER 8
#define NDMAX 0
#define BDELAY 40
#define PACE 110
constexpr unsigned long long DEADLINE = 50000000ull;

struct Args { const char* x; const char* W; char* out; char* ws[NR]; int rank; };

__host__ __device__ __forceinline__ int lds_byte(int r, int c) { const int st = (r >> 4) * 2 + (c >> 5), rr = r & 15, cc = c & 31, ob = rr * 64 + cc * 2; return st * 1024 + (ob ^ (((ob >> 9) & 1) << 5)); }
__host__ __device__ __forceinline__ void stage_rc(int b, int& R, int& C) { const int st = b / 1024, sb = b % 1024, swz = sb ^ (((sb >> 9) & 1) << 5); R = (st >> 1) * 16 + swz / 64; C = (st & 1) * 32 + (swz % 64) / 2; }
__host__ __device__ __forceinline__ int fk(int k) { return (k & 1) | ((k & 2) << 1) | (k & 8); }

__device__ __forceinline__ unsigned cvt_pk_bf16(float lo, float hi) { unsigned r; asm("v_cvt_pk_bf16_f32 %0, %1, %2" : "=v"(r) : "v"(lo), "v"(hi)); return r; }
__device__ __forceinline__ unsigned gelu2(f32x2 y) {
  const float k1 = -2.0f * 0.7978845608028654f * 1.4426950408889634f, k2 = k1 * 0.044715f;
  const f32x2 z = y * __builtin_elementwise_fma(y * y, (f32x2){k2, k2}, (f32x2){k1, k1});
  f32x2 e; e.x = __builtin_amdgcn_exp2f(z.x); e.y = __builtin_amdgcn_exp2f(z.y);
  const f32x2 d = e + 1.0f;
  f32x2 r; r.x = __builtin_amdgcn_rcpf(d.x); r.y = __builtin_amdgcn_rcpf(d.y);
  const f32x2 o = y * r;
  return cvt_pk_bf16(o.x, o.y);
}
template <int AUX> __device__ __forceinline__ void st128(u32x4 v, __amdgpu_buffer_rsrc_t r, unsigned voff, unsigned soff) {
  __builtin_amdgcn_raw_buffer_store_b128(v, r, voff, soff, AUX);
  asm volatile("s_nop 1" :: "v"(v) : "memory");
}
__device__ __forceinline__ __amdgpu_buffer_rsrc_t mk_rsrc(const void* p, unsigned bytes) { return __builtin_amdgcn_make_buffer_rsrc((void*)p, (short)0, (int)bytes, 0x00020000); }

__device__ __forceinline__ void spin_flags(const unsigned* f, unsigned need, unsigned* tmo, unsigned call, LAS unsigned* dead) {
  const unsigned long long t0 = __builtin_amdgcn_s_memrealtime();
  __builtin_amdgcn_s_waitcnt(0xC07F);
  for (;;) {
    const unsigned v = __hip_atomic_load(f, __ATOMIC_RELAXED, POLL_SCOPE);
    if (__all((int)(v - need) >= 0)) break;
    if (__builtin_amdgcn_readfirstlane(__hip_atomic_load(tmo, __ATOMIC_RELAXED, __HIP_MEMORY_SCOPE_AGENT)) == call ||
        __builtin_amdgcn_s_memrealtime() - t0 > DEADLINE) {
      __hip_atomic_store(tmo, call, __ATOMIC_RELAXED, __HIP_MEMORY_SCOPE_AGENT);
      *dead = 1u;
      asm volatile("s_waitcnt vmcnt(0) lgkmcnt(0)" ::: "memory");
      break;
    }
    __builtin_amdgcn_s_sleep(1);
  }
  asm volatile("" ::: "memory");
}

__global__ __launch_bounds__(512, 2) void ag_gemm_kernel(const Args a) {
  extern __shared__ __attribute__((aligned(16))) unsigned char shm_[];
  LAS unsigned char* lds = (LAS unsigned char*)shm_;
  const int tid = threadIdx.x, wid = __builtin_amdgcn_readfirstlane(tid >> 6), lane = tid & 63, wr = wid >> 2, wc = wid & 3, fr = lane & 15, fq = lane >> 4;
  const int bid = blockIdx.x, rank = a.rank;
  char* const ws = a.ws[0];
  unsigned prev;
  asm volatile("s_load_dword %0, %1, 0x0" : "=s"(prev) : "s"(ws + WS_CTR + (size_t)bid * 128) : "memory");
  const int xcd = bid & 7, idx = bid >> 3;
  const int pm = (xcd & 1) * 8 + (idx & 7), pn = (xcd >> 1) * 4 + (idx >> 3);
  const int unit = (bid & 7) | ((bid >> 6) << 3);
  char* const wsd = a.ws[(bid >> 3) & (NR - 1)];
  const __amdgpu_buffer_rsrc_t srdB = mk_rsrc(a.W, W_BYTES);
  const __amdgpu_buffer_rsrc_t srdX = mk_rsrc(a.x, X_BYTES);
  const __amdgpu_buffer_rsrc_t srdFd = mk_rsrc(wsd + WS_FLAG, FLAG_BYTES);
  const __amdgpu_buffer_rsrc_t srdFl = mk_rsrc(ws + WS_FLAG, FLAG_BYTES);
  const unsigned ldsb = (unsigned)(size_t)lds;
  const unsigned stg = (unsigned)L_STG + (unsigned)wid * 1024u;
  const int dph = ((unit & 7) * DPER) >> 3;
#define NDBL(t) ((((t) + dph + DPER) / DPER - 1) < NDMAX ? ((t) + dph < 0 ? 0 : ((t) + dph) / DPER) : NDMAX)
#define KP(t) (LEAD + (t) + NDBL(t))
  unsigned vx1;
  {
    int R, C; stage_rc((unit & 7) * 2048 + wc * 512 + (lane & 31) * 16, R, C);
    const unsigned src = (unsigned)(((unit >> 3) * 128 + R) * KT + C) * 2u + (unsigned)(lane >> 5) * 128u;
    vx1 = !wr ? src : FSTR == 4 ? (lane < 4 ? (unsigned)(pm >> 1) * FSRC + (unsigned)((pm & 1) * 64 + lane * 16) : OOB)
                            : (lane < 16 ? (unsigned)(pm >> 1) * FSRC + (unsigned)((pm & 1) * 16 + lane) * FSTR : OOB);
  }
#define XLOAD(kx, sl) asm volatile("s_mov_b32 m0, %0\n\ts_nop 0\n\tbuffer_load_dwordx4 %1, %2, %3 offen lds" \
        :: "s"(ldsb + stg + (sl) * 8192), "v"(vx1), "s"(srdX), "s"((unsigned)(kx) << 7) : "m0", "memory")
#define POLL(kp) asm volatile("s_mov_b32 m0, %0\n\ts_nop 0\n\tbuffer_load_dwordx4 %1, %2, %3 offen" POLL_SC " lds" \
        :: "s"(ldsb + stg), "v"(vx1), "s"(srdFl), "s"((unsigned)((kp) & 7) * FSLOT) : "m0", "memory")
#define PLOAD(voff, kx, i) asm volatile("s_mov_b32 m0, %0\n\ts_nop 0\n\tbuffer_load_dwordx4 %1, %2, %3 offen lds" \
        :: "s"(ldsb + L_PRO + wc * 4096 + (i) * 1024), "v"(voff), "s"(srdX), "s"((unsigned)(kx) << 7) : "m0", "memory")
#define PREAD(i) (*(const LAS u32x4*)(lds + L_PRO + wc * 4096 + (i) * 1024 + lane * 16))
  LAS unsigned* const dead = (LAS unsigned*)(lds + L_DEAD);
  unsigned vs0 = 0;
  if (!wr) {
    int R, C; stage_rc((unit & 7) * 2048 + lane * 16, R, C);
    const unsigned s0 = (unsigned)(((unit >> 3) * 128 + R) * KT + C) * 2u;
    PLOAD(s0, wc, 0); PLOAD(s0 + 64u, wc, 1);
    PLOAD(vx1, LEAD - 1, 2);
    XLOAD(KP(0), 0); XLOAD(KP(1), 1);
    vs0 = s0;
  } else if (tid == 256) *dead = 0u;
  asm volatile("s_waitcnt lgkmcnt(0)" : "+s"(prev) :: "memory");
  const unsigned call = prev + 1u, base = prev * (unsigned)NKT, par = call & 1u;
  const unsigned long long t_start = __builtin_amdgcn_s_memrealtime();
  const __amdgpu_buffer_rsrc_t srdA = mk_rsrc(ws + WS_IMG + (size_t)par * IMG_BYTES, IMG_BYTES);
  const __amdgpu_buffer_rsrc_t srdD = mk_rsrc(wsd + WS_IMG + (size_t)par * IMG_BYTES, IMG_BYTES);
  unsigned* const tmo = (unsigned*)(ws + WS_TMO);
  const unsigned* const myflags = (const unsigned*)(ws + WS_FLAG + (unsigned)(pm >> 1) * FSRC + (unsigned)((pm & 1) * 16 + (lane & 15)) * FSTR);

  const unsigned vA = (unsigned)(wc * 4096 + lane * 16);
  const unsigned vB0 = (unsigned)(wc * 16 + (lane >> 4)) * (NL * 2) + (unsigned)(((lane & 15) ^ fk(lane >> 4)) * 16);
  const unsigned vB1 = vB0 + 4 * (NL * 2) - 1024, vB2 = (vB0 ^ 128u) + 8 * (NL * 2) - 2048, vB3 = (vB0 ^ 128u) + 12 * (NL * 2) - 3072;
  const unsigned ldq = ldsb + (unsigned)wc * 4096u;
  const int aoff = lds_byte(wr * 64 + fr, fq * 8);
  const int q_ = (lane & 15) >> 2, p_ = lane & 3;
  const int f_ = (q_ & 1) | ((q_ & 2) << 1) | ((fq & 1) << 3);
  int boff0 = 4 * HTB + 256 * (8 * fq + q_) + 16 * ((4 * wc + p_) ^ f_) + 8 * (p_ & 1);
  int boff1 = boff0 ^ 8;
  asm volatile("" : "+v"(boff0), "+v"(boff1));
  const unsigned sA0 = (unsigned)(2 * pm) * HTB, sB0 = (unsigned)pn * 512u;
  const unsigned sD0 = (unsigned)(rank * 4 + (unit >> 3)) * HTB + (unsigned)(unit & 7) * 2048u;
  const int stgl = (int)stg + lane * 16, chkl = (int)stg + (lane & 15) * (FSTR == 4 ? 4 : 16);
  const unsigned vS2 = (unsigned)(wc * 512 + (lane & 31) * 16) + (unsigned)(lane >> 5) * KSTEP;
  const unsigned vS1 = lane < 32 ? vS2 : OOB;
  const unsigned vF = (unsigned)rank * FSRC + (unsigned)unit * FSTR;

#define SA(b, h) (((b) * 2 + (h)) * HTB)
#define SB(b, h) ((4 + (b) * 2 + (h)) * HTB)
#define PA_(b, h, kt, i) asm volatile("s_mov_b32 m0, %0\n\ts_nop 0\n\tbuffer_load_dwordx4 %1, %2, %3 offen offset:" #i " sc1 lds" \
    :: "s"(ldq + SA(b, h)), "v"(vA), "s"(srdA), "s"(sA0 + (unsigned)(kt) * KSTEP + (h) * HTB) : "m0", "memory")
#define PB_(b, h, kt, v, i) asm volatile("s_mov_b32 m0, %0\n\ts_nop 0\n\tbuffer_load_dwordx4 %1, %2, %3 offen offset:" #i " lds" \
    :: "s"(ldq + SB(b, h)), "v"(v), "s"(srdB), "s"(sB0 + (unsigned)(kt) * KSTEP + (h) * 256) : "m0", "memory")
#define STAGE_A(b, h, kt) do { PA_(b, h, kt, 0); PA_(b, h, kt, 1024); PA_(b, h, kt, 2048); PA_(b, h, kt, 3072); } while (0)
#define STAGE_B(b, h, kt) do { PB_(b, h, kt, vB0, 0); PB_(b, h, kt, vB1, 1024); PB_(b, h, kt, vB2, 2048); PB_(b, h, kt, vB3, 3072); } while (0)
#define TR(off) __builtin_amdgcn_ds_read_tr16_b64_v4i16((LAS s16x4*)(lds + (off)))
#define LDA2(dst, b, h, m0_) do { _Pragma("unroll") for (int m = m0_; m < m0_ + 2; ++m) _Pragma("unroll") for (int k = 0; k < 2; ++k) dst[m][k] = *(const LAS bf16x8*)(lds + SA(b, h) + aoff + m * 2048 + k * 1024); } while (0)
#define LDB1(dst, b, h, k) do { \
    { s16x4 lo = TR(boff0 + ((b) * 2 + (h)) * HTB + k * 8192), hi = TR(boff0 + ((b) * 2 + (h)) * HTB + k * 8192 + 1024); dst[0][k] = __builtin_shufflevector(lo, hi, 0, 1, 2, 3, 4, 5, 6, 7); } \
    { s16x4 lo = TR(boff1 + ((b) * 2 + (h)) * HTB + k * 8192), hi = TR(boff1 + ((b) * 2 + (h)) * HTB + k * 8192 + 1024); dst[1][k] = __builtin_shufflevector(lo, hi, 0, 1, 2, 3, 4, 5, 6, 7); } } while (0)
#define WAIT_V(n) asm volatile("s_waitcnt vmcnt(" #n ")" ::: "memory")
#define WAIT_L(n) do { asm volatile("" ::: "memory"); __builtin_amdgcn_s_waitcnt(0xC07F | ((n) << 8)); asm volatile("" ::: "memory"); } while (0)
#define BAR __builtin_amdgcn_s_barrier()
#define SCHED __builtin_amdgcn_sched_barrier(0)
#define MMA(ai, bj, At, Bt) do { _Pragma("unroll") for (int n = 0; n < 2; ++n) _Pragma("unroll") for (int m_ = 0; m_ < 4; ++m_) _Pragma("unroll") for (int k_ = 0; k_ < 2; ++k_) { \
    const int m = n ? 3 - m_ : m_, k = k_ ^ (m_ & 1); \
    acc[ai][bj][m][n] = __builtin_amdgcn_mfma_f32_16x16x32_bf16(Bt[n][k], At[m][k], acc[ai][bj][m][n], 0, 0, 0); SCHED; } } while (0)
#define MMA2(ai, At, B0, B1) do { SCHED; MMA(ai, 0, At, B0); MMA(ai, 1, At, B1); SCHED; } while (0)
#define STAMP(i) do {} while (0)
#define STAMPC(i) do {} while (0)
#define STAMPT(t) do {} while (0)
#define SLOWCNT do {} while (0)
#define PUSH_RD(sl) (*(const LAS u32x4*)(lds + stgl + (sl) * 8192))
#define PUBLISH(val, voff, soff) do { __builtin_amdgcn_raw_buffer_store_b32(base + (unsigned)(val), srdFd, voff, soff, 17); asm volatile("" ::: "memory"); } while (0)
#define CHECK_ON true
#define CHECK(fv, kp) do { if (__builtin_expect(CHECK_ON && (kp) < NKT && !__all((int)((fv) - (base + (unsigned)(kp) + 1u)) >= 0), 0)) { \
        SLOWCNT; spin_flags(myflags + ((kp) & 7) * (FSLOT / 4), base + (unsigned)(kp) + 1u, tmo, call, dead); } } while (0)
#define PACE_UNTIL(ticks) do { while (__builtin_amdgcn_s_memrealtime() - t_start < (unsigned long long)(ticks)) __builtin_amdgcn_s_sleep(1); } while (0)

  f32x4 acc[2][2][4][2];
#pragma unroll
  for (int i = 0; i < 2; ++i)
#pragma unroll
    for (int j = 0; j < 2; ++j)
#pragma unroll
      for (int m = 0; m < 4; ++m)
#pragma unroll
        for (int n = 0; n < 2; ++n) acc[i][j][m][n] = (f32x4){0.f, 0.f, 0.f, 0.f};
  bf16x8 At[4][2], B0a[2][2], B0b[2][2], B1[2][2];
  if (!wr) {
    if (wc < NPRO) {
      WAIT_V(3);
      const u32x4 pv0 = PREAD(0), pv1 = PREAD(1);
      PACE_UNTIL(wc * PACE);
      st128<17>(pv0, srdD, (unsigned)lane * 16u, sD0 + (unsigned)wc * KSTEP);
      st128<17>(pv1, srdD, (unsigned)lane * 16u + 1024u, sD0 + (unsigned)wc * KSTEP);
      WAIT_V(0);
      PUBLISH(wc + 1, lane == 0 ? vF : OOB, (unsigned)wc * FSLOT);
      if (wc + 4 < NPRO) {
        PLOAD(vs0, wc + 4, 0); PLOAD(vs0 + 64u, wc + 4, 1);
        WAIT_V(0);
        const u32x4 pw0 = PREAD(0), pw1 = PREAD(1);
        PACE_UNTIL((wc + 4) * PACE);
        st128<17>(pw0, srdD, (unsigned)lane * 16u, sD0 + (unsigned)(wc + 4) * KSTEP);
        st128<17>(pw1, srdD, (unsigned)lane * 16u + 1024u, sD0 + (unsigned)(wc + 4) * KSTEP);
        WAIT_V(0);
        PUBLISH(wc + 5, lane == 0 ? vF : OOB, (unsigned)(wc + 4) * FSLOT);
      }
    } else WAIT_V(0);
    STAMP(1);
    { const u32x4 pv2 = PREAD(2);
      PACE_UNTIL(NPRO * PACE);
      st128<17>(pv2, srdD, vS1, sD0 + (unsigned)(LEAD - 1) * KSTEP); }
    WAIT_V(0);
    WAIT_L(0);
    STAMP(2);
    BAR;
    LDB1(B0a, 0, 0, 0); LDB1(B0a, 0, 0, 1);
    WAIT_L(0);
    BAR;
    STAMP(5); STAMPC(44);
    unsigned vF1 = tid == 0 ? vF : OOB, vF2 = tid < 2 ? vF : OOB;
    asm volatile("" : "+v"(vF1), "+v"(vF2));
    u32x4 pv = PUSH_RD(0);
    unsigned fval, foff, soff;
#define PUB_PREP(t) do { const int kf_ = KP((t) - PDEL); fval = base + (unsigned)(kf_ + 1 + lane); \
      foff = ((KP((t) - PDEL + 1) - kf_ == 2 && kf_ + 1 < NKT) ? vF2 : vF1) + (unsigned)((kf_ + lane) & 7) * FSLOT + (kf_ < NKT ? 0u : OOB); \
      soff = KP((t) + 1) - KP(t) == 2 ? vS2 : vS1; asm volatile("" : "+v"(fval), "+v"(foff), "+v"(soff)); } while (0)
    PUB_PREP(0);
#define WAIT_MB WAIT_V(9)
#define KTILE0(t, b, B0c, B0n) do { \
    __builtin_amdgcn_raw_buffer_store_b32(fval, srdFd, foff, 0u, 17); asm volatile("" ::: "memory"); \
    st128<17>(pv, srdD, soff, sD0 + (unsigned)KP(t) * KSTEP); \
    LDB1(B1, b, 1, 0); PB_(b, 0, (t) + 2, vB0, 0); LDB1(B1, b, 1, 1); PB_(b, 0, (t) + 2, vB1, 1024); \
    LDA2(At, b, 0, 0); PB_(b, 0, (t) + 2, vB2, 2048); LDA2(At, b, 0, 2); PB_(b, 0, (t) + 2, vB3, 3072); \
    WAIT_L(0); BAR; MMA2(0, At, B0c, B1); WAIT_V(10); BAR; SCHED; \
    XLOAD(KP((t) + 2), b); pv = PUSH_RD((b) ^ 1); \
    LDA2(At, b, 1, 0); PB_(b, 1, (t) + 2, vB0, 0); LDA2(At, b, 1, 2); PB_(b, 1, (t) + 2, vB1, 1024); \
    PUB_PREP((t) + 1); \
    LDB1(B0n, (b) ^ 1, 0, 0); PB_(b, 1, (t) + 2, vB2, 2048); LDB1(B0n, (b) ^ 1, 0, 1); PB_(b, 1, (t) + 2, vB3, 3072); \
    WAIT_L(0); BAR; MMA2(1, At, B0c, B1); WAIT_MB; BAR; SCHED; } while (0)
    for (int t = 0; t < NKT; t += 2) {
      KTILE0(t, 0, B0a, B0b);
      KTILE0(t + 1, 1, B0b, B0a);
      STAMPT(t);
    }
    STAMPC(45);
    BAR;
  } else {
    PACE_UNTIL(BDELAY);
    STAGE_B(0, 0, 0); STAGE_B(0, 1, 0); STAGE_B(1, 0, 1); STAGE_B(1, 1, 1);
    STAMP(1);
    spin_flags(myflags, base + 1u, tmo, call, dead);
    STAMP(2);
    STAGE_A(0, 0, 0); STAGE_A(0, 1, 0);
    spin_flags(myflags + FSLOT / 4, base + 2u, tmo, call, dead);
    STAMP(3);
    WAIT_V(0);
    STAGE_A(1, 0, 1);
    STAMP(4);
    BAR;
    LDB1(B0a, 0, 0, 0); LDB1(B0a, 0, 0, 1);
    WAIT_L(0);
    BAR;
    STAMP(5); STAMPC(44);
    BAR;
#define KTILE1(t, b, B0c, B0n) do { \
    POLL((t) + 2); \
    PA_((b) ^ 1, 1, (t) + 1, 0); LDB1(B1, b, 1, 0); PA_((b) ^ 1, 1, (t) + 1, 1024); LDB1(B1, b, 1, 1); \
    PA_((b) ^ 1, 1, (t) + 1, 2048); LDA2(At, b, 0, 0); PA_((b) ^ 1, 1, (t) + 1, 3072); LDA2(At, b, 0, 2); \
    WAIT_V(9); WAIT_L(0); BAR; MMA2(0, At, B0c, B1); BAR; SCHED; \
    { WAIT_V(4); const unsigned fv_ = *(const LAS unsigned*)(lds + chkl); asm volatile("" ::: "memory"); \
      LDA2(At, b, 1, 0); \
      CHECK(fv_, (t) + 2); } \
    PA_(b, 0, (t) + 2, 0); LDA2(At, b, 1, 2); PA_(b, 0, (t) + 2, 1024); LDB1(B0n, (b) ^ 1, 0, 0); \
    PA_(b, 0, (t) + 2, 2048); LDB1(B0n, (b) ^ 1, 0, 1); PA_(b, 0, (t) + 2, 3072); \
    WAIT_L(0); BAR; MMA2(1, At, B0c, B1); BAR; SCHED; } while (0)
    for (int t = 0; t < NKT; t += 2) {
      KTILE1(t, 0, B0a, B0b);
      KTILE1(t + 1, 1, B0b, B0a);
      STAMPT(t);
    }
    STAMPC(45);
  }
  {
    int tz = threadIdx.x; asm volatile("" : "+v"(tz));
    const int lane2 = tz & 63, fr2 = lane2 & 15, fq2 = lane2 >> 4;
    const bool odd = fq2 & 1;
    const bool bad = __builtin_amdgcn_readfirstlane(*dead) != 0u;
    const __amdgpu_buffer_rsrc_t srdO = mk_rsrc(a.out, (unsigned)MT * NL * 2);
    const unsigned ooff = ((unsigned)(pm * BM + wr * 64 + fr2) * NL + (unsigned)(pn * BM + wc * 32 + 8 * fq2)) * 2u;
    if (bad) {
#pragma unroll
      for (int i = 0; i < 2; ++i)
#pragma unroll
        for (int j = 0; j < 2; ++j)
#pragma unroll
          for (int m = 0; m < 4; ++m)
#pragma unroll
            for (int n = 0; n < 2; ++n) acc[i][j][m][n] = (f32x4){__builtin_nanf(""), __builtin_nanf(""), __builtin_nanf(""), __builtin_nanf("")};
    }
#pragma unroll
    for (int ai = 0; ai < 2; ++ai)
#pragma unroll
      for (int m = 0; m < 4; ++m)
#pragma unroll
        for (int bj = 0; bj < 2; ++bj) {
          const f32x4 v0 = acc[ai][bj][m][0], v1 = acc[ai][bj][m][1];
          const unsigned a0 = gelu2((f32x2){v0[0], v0[1]}), a1 = gelu2((f32x2){v0[2], v0[3]});
          const unsigned b0 = gelu2((f32x2){v1[0], v1[1]}), b1 = gelu2((f32x2){v1[2], v1[3]});
          u32x4 w; w.x = odd ? b0 : a0; w.y = odd ? b1 : a1; w.z = odd ? a0 : b0; w.w = odd ? a1 : b1;
          st128<16>(w, srdO, ooff + bj * (HALF * 2), (unsigned)(ai * HALF + m * 16) * (NL * 2));
        }
    STAMP(40);
    if (tz == 0) __hip_atomic_store((unsigned*)(ws + WS_CTR + (size_t)bid * 128), call, __ATOMIC_RELAXED, __HIP_MEMORY_SCOPE_AGENT);
  }
}
}

extern "C" void kernel_launch(int rank, int nranks, ncclComm_t comm,
                              hipStream_t stream,
                              void* const* d_in, const int* in_sizes, int n_in,
                              void* d_out, int out_size,
                              const int* in_dtypes, int out_dtype,
                              void* const* d_out_all,
                              void* const* d_ws_all, size_t ws_bytes) {
  using namespace agg;
  if (nranks != NR || (unsigned)rank >= (unsigned)NR || n_in < 2 || ws_bytes < WS_NEED || in_sizes[0] != MS * KT || in_sizes[1] != KT * NL || out_size != MT * NL) {
    fprintf(stderr, "kernel_launch: unexpected problem shape\n");
    return;
  }
  static unsigned attr_done = 0;
  int dev = 0; HIP_CHECK(hipGetDevice(&dev));
  if (!(attr_done >> (dev & 31) & 1u)) {
    HIP_CHECK(hipFuncSetAttribute((const void*)ag_gemm_kernel, hipFuncAttributeMaxDynamicSharedMemorySize, LDS_BYTES));
    attr_done |= 1u << (dev & 31);
  }
  Args a;
  a.x = (const char*)d_in[0]; a.W = (const char*)d_in[1]; a.out = (char*)d_out; a.rank = rank;
  for (int p = 0; p < NR; ++p) a.ws[p] = (char*)d_ws_all[(rank + p) & (NR - 1)];
  hipLaunchKernelGGL(ag_gemm_kernel, dim3(256), dim3(512), LDS_BYTES, stream, a);
  HIP_CHECK(hipGetLastError());
}
```

```cpp
#include <hip/hip_runtime.h>
#include <rccl/rccl.h>
#include <cstdio>

#define HIP_CHECK(expr) do { hipError_t _e = (expr); if (_e != hipSuccess) { \
    fprintf(stderr, "HIP error %s at %s:%d\n", hipGetErrorString(_e), __FILE__, __LINE__); \
    return; } } while (0)

namespace agg {
#define LAS __attribute__((address_space(3)))
typedef short bf16x8 __attribute__((ext_vector_type(8)));
typedef short s16x4 __attribute__((ext_vector_type(4)));
typedef float f32x4 __attribute__((ext_vector_type(4)));
typedef float f32x2 __attribute__((ext_vector_type(2)));
typedef unsigned u32x4 __attribute__((ext_vector_type(4)));

constexpr int NR = 8;
constexpr int MT = 4096, KT = 4096, NL = 4096;
constexpr int MS = MT / NR;
constexpr int BM = 256, BK = 64, HALF = 128, HTB = 16384, NKT = KT / BK;
constexpr unsigned KSTEP = 32u * HTB;
static_assert(KSTEP == 64u * NL * 2u, "");
constexpr unsigned IMG_BYTES = (unsigned)MT * KT * 2;
constexpr unsigned X_BYTES = (unsigned)MS * KT * 2, W_BYTES = (unsigned)KT * NL * 2;
constexpr size_t WS_CTR = 0;
constexpr size_t WS_TMO = 32768;
#define FSTR 16
constexpr size_t WS_FLAG = 65536;
constexpr unsigned FSLOT = 32 * FSTR, FSRC = 8 * FSLOT, FLAG_BYTES = NR * FSRC;
static_assert((FSTR == 4 || FSTR >= 16) && WS_FLAG + FLAG_BYTES <= 524288, "");
#define POLL_SYS 1
#define POLL_SC " sc0 sc1"
#define POLL_SCOPE __HIP_MEMORY_SCOPE_SYSTEM
constexpr size_t WS_STAMP = 524288;
constexpr size_t WS_IMG = 1u << 20;
constexpr size_t WS_NEED = WS_IMG + 2 * (size_t)IMG_BYTES;
constexpr int L_STG = 8 * HTB;
constexpr int L_DEAD = L_STG + 8192 + 4096;
constexpr int L_PRO = L_STG + 16384;
constexpr int LDS_BYTES = L_PRO + 16384;
static_assert(LDS_BYTES <= 160 * 1024, "");
constexpr unsigned OOB = 0x40000000u;
#define LEAD 4
#define NPRO (LEAD - 1)
static_assert(NPRO <= 8, "");
#define TAIL 0
#define ONEPOLL 1
#define ALATE 1
#define PDEL 1
#define DPER 8
#define NDMAX 0
#define BDELAY 40
#define PACE 110
constexpr unsigned long long DEADLINE = 50000000ull;

struct Args { const char* x; const char* W; char* out; char* ws[NR]; int rank; };

__host__ __device__ __forceinline__ int lds_byte(int r, int c) { const int st = (r >> 4) * 2 + (c >> 5), rr = r & 15, cc = c & 31, ob = rr * 64 + cc * 2; return st * 1024 + (ob ^ (((ob >> 9) & 1) << 5)); }
__host__ __device__ __forceinline__ void stage_rc(int b, int& R, int& C) { const int st = b / 1024, sb = b % 1024, swz = sb ^ (((sb >> 9) & 1) << 5); R = (st >> 1) * 16 + swz / 64; C = (st & 1) * 32 + (swz % 64) / 2; }
__host__ __device__ __forceinline__ int fk(int k) { return (k & 1) | ((k & 2) << 1) | (k & 8); }

__device__ __forceinline__ unsigned cvt_pk_bf16(float lo, float hi) { unsigned r; asm("v_cvt_pk_bf16_f32 %0, %1, %2" : "=v"(r) : "v"(lo), "v"(hi)); return r; }
__device__ __forceinline__ unsigned gelu2(f32x2 y) {
  const float k1 = -2.0f * 0.7978845608028654f * 1.4426950408889634f, k2 = k1 * 0.044715f;
  const f32x2 z = y * __builtin_elementwise_fma(y * y, (f32x2){k2, k2}, (f32x2){k1, k1});
  f32x2 e; e.x = __builtin_amdgcn_exp2f(z.x); e.y = __builtin_amdgcn_exp2f(z.y);
  const f32x2 d = e + 1.0f;
  f32x2 r; r.x = __builtin_amdgcn_rcpf(d.x); r.y = __builtin_amdgcn_rcpf(d.y);
  const f32x2 o = y * r;
  return cvt_pk_bf16(o.x, o.y);
}
template <int AUX> __device__ __forceinline__ void st128(u32x4 v, __amdgpu_buffer_rsrc_t r, unsigned voff, unsigned soff) {
  __builtin_amdgcn_raw_buffer_store_b128(v, r, voff, soff, AUX);
  asm volatile("s_nop 1" :: "v"(v) : "memory");
}
__device__ __forceinline__ __amdgpu_buffer_rsrc_t mk_rsrc(const void* p, unsigned bytes) { return __builtin_amdgcn_make_buffer_rsrc((void*)p, (short)0, (int)bytes, 0x00020000); }

__device__ __forceinline__ void spin_flags(const unsigned* f, unsigned need, unsigned* tmo, unsigned call, LAS unsigned* dead) {
  const unsigned long long t0 = __builtin_amdgcn_s_memrealtime();
  __builtin_amdgcn_s_waitcnt(0xC07F);
  for (;;) {
    const unsigned v = __hip_atomic_load(f, __ATOMIC_RELAXED, POLL_SCOPE);
    if (__all((int)(v - need) >= 0)) break;
    if (__builtin_amdgcn_readfirstlane(__hip_atomic_load(tmo, __ATOMIC_RELAXED, __HIP_MEMORY_SCOPE_AGENT)) == call ||
        __builtin_amdgcn_s_memrealtime() - t0 > DEADLINE) {
      __hip_atomic_store(tmo, call, __ATOMIC_RELAXED, __HIP_MEMORY_SCOPE_AGENT);
      *dead = 1u;
      asm volatile("s_waitcnt vmcnt(0) lgkmcnt(0)" ::: "memory");
      break;
    }
    __builtin_amdgcn_s_sleep(1);
  }
  asm volatile("" ::: "memory");
}

__global__ __launch_bounds__(512, 2) void ag_gemm_kernel(const Args a) {
  extern __shared__ __attribute__((aligned(16))) unsigned char shm_[];
  LAS unsigned char* lds = (LAS unsigned char*)shm_;
  const int tid = threadIdx.x, wid = __builtin_amdgcn_readfirstlane(tid >> 6), lane = tid & 63, wr = wid >> 2, wc = wid & 3, fr = lane & 15, fq = lane >> 4;
  const int bid = blockIdx.x, rank = a.rank;
  char* const ws = a.ws[0];
  unsigned prev;
  asm volatile("s_load_dword %0, %1, 0x0" : "=s"(prev) : "s"(ws + WS_CTR + (size_t)bid * 128) : "memory");
  const int xcd = bid & 7, idx = bid >> 3;
  const int pm = (xcd & 1) * 8 + (idx & 7), pn = (xcd >> 1) * 4 + (idx >> 3);
  const int unit = (bid & 7) | ((bid >> 6) << 3);
  char* const wsd = a.ws[(bid >> 3) & (NR - 1)];
  const __amdgpu_buffer_rsrc_t srdB = mk_rsrc(a.W, W_BYTES);
  const __amdgpu_buffer_rsrc_t srdX = mk_rsrc(a.x, X_BYTES);
  const __amdgpu_buffer_rsrc_t srdFd = mk_rsrc(wsd + WS_FLAG, FLAG_BYTES);
  const __amdgpu_buffer_rsrc_t srdFl = mk_rsrc(ws + WS_FLAG, FLAG_BYTES);
  const unsigned ldsb = (unsigned)(size_t)lds;
  const unsigned stg = (unsigned)L_STG + (unsigned)wid * 1024u;
  const int dph = ((unit & 7) * DPER) >> 3;
#define NDBL(t) ((((t) + dph + DPER) / DPER - 1) < NDMAX ? ((t) + dph < 0 ? 0 : ((t) + dph) / DPER) : NDMAX)
#define KP(t) (LEAD + (t) + NDBL(t))
  unsigned vx1;
  {
    int R, C; stage_rc((unit & 7) * 2048 + wc * 512 + (lane & 31) * 16, R, C);
    const unsigned src = (unsigned)(((unit >> 3) * 128 + R) * KT + C) * 2u + (unsigned)(lane >> 5) * 128u;
    vx1 = !wr ? src : FSTR == 4 ? (lane < 4 ? (unsigned)(pm >> 1) * FSRC + (unsigned)((pm & 1) * 64 + lane * 16) : OOB)
                            : (lane < 16 ? (unsigned)(pm >> 1) * FSRC + (unsigned)((pm & 1) * 16 + lane) * FSTR : OOB);
    if (ONEPOLL && wid > 4) vx1 = OOB;
  }
#define XLOAD(kx, sl) asm volatile("s_mov_b32 m0, %0\n\ts_nop 0\n\tbuffer_load_dwordx4 %1, %2, %3 offen lds" \
        :: "s"(ldsb + stg + (sl) * 8192), "v"(vx1), "s"(srdX), "s"((unsigned)(kx) << 7) : "m0", "memory")
#define POLL(kp) asm volatile("s_mov_b32 m0, %0\n\ts_nop 0\n\tbuffer_load_dwordx4 %1, %2, %3 offen" POLL_SC " lds" \
        :: "s"(ldsb + stg), "v"(vx1), "s"(srdFl), "s"((unsigned)((kp) & 7) * FSLOT) : "m0", "memory")
#define PLOAD(voff, kx, i) asm volatile("s_mov_b32 m0, %0\n\ts_nop 0\n\tbuffer_load_dwordx4 %1, %2, %3 offen lds" \
        :: "s"(ldsb + L_PRO + wc * 4096 + (i) * 1024), "v"(voff), "s"(srdX), "s"((unsigned)(kx) << 7) : "m0", "memory")
#define PREAD(i) (*(const LAS u32x4*)(lds + L_PRO + wc * 4096 + (i) * 1024 + lane * 16))
  LAS unsigned* const dead = (LAS unsigned*)(lds + L_DEAD);
  unsigned vs0 = 0;
  if (!wr) {
    int R, C; stage_rc((unit & 7) * 2048 + lane * 16, R, C);
    const unsigned s0 = (unsigned)(((unit >> 3) * 128 + R) * KT + C) * 2u;
    PLOAD(s0, wc, 0); PLOAD(s0 + 64u, wc, 1);
    PLOAD(vx1, LEAD - 1, 2);
    XLOAD(KP(0), 0); XLOAD(KP(1), 1);
    vs0 = s0;
  } else if (tid == 256) { dead[0] = 0u; dead[1] = 0u; }
  asm volatile("s_waitcnt lgkmcnt(0)" : "+s"(prev) :: "memory");
  const unsigned call = prev + 1u, base = prev * (unsigned)NKT, par = call & 1u;
  const unsigned long long t_start = __builtin_amdgcn_s_memrealtime();
  const __amdgpu_buffer_rsrc_t srdA = mk_rsrc(ws + WS_IMG + (size_t)par * IMG_BYTES, IMG_BYTES);
  const __amdgpu_buffer_rsrc_t srdD = mk_rsrc(wsd + WS_IMG + (size_t)par * IMG_BYTES, IMG_BYTES);
  unsigned* const tmo = (unsigned*)(ws + WS_TMO);
  const unsigned* const myflags = (const unsigned*)(ws + WS_FLAG + (unsigned)(pm >> 1) * FSRC + (unsigned)((pm & 1) * 16 + (lane & 15)) * FSTR);

  const unsigned vA = (unsigned)(wc * 4096 + lane * 16);
  const unsigned vB0 = (unsigned)(wc * 16 + (lane >> 4)) * (NL * 2) + (unsigned)(((lane & 15) ^ fk(lane >> 4)) * 16);
  const unsigned vB1 = vB0 + 4 * (NL * 2) - 1024, vB2 = (vB0 ^ 128u) + 8 * (NL * 2) - 2048, vB3 = (vB0 ^ 128u) + 12 * (NL * 2) - 3072;
  const unsigned ldq = ldsb + (unsigned)wc * 4096u;
  const int aoff = lds_byte(wr * 64 + fr, fq * 8);
  const int q_ = (lane & 15) >> 2, p_ = lane & 3;
  const int f_ = (q_ & 1) | ((q_ & 2) << 1) | ((fq & 1) << 3);
  int boff0 = 4 * HTB + 256 * (8 * fq + q_) + 16 * ((4 * wc + p_) ^ f_) + 8 * (p_ & 1);
  int boff1 = boff0 ^ 8;
  asm volatile("" : "+v"(boff0), "+v"(boff1));
  const unsigned sA0 = (unsigned)(2 * pm) * HTB, sB0 = (unsigned)pn * 512u;
  const unsigned sD0 = (unsigned)(rank * 4 + (unit >> 3)) * HTB + (unsigned)(unit & 7) * 2048u;
  const int stgl = (int)stg + lane * 16, chkl = (ONEPOLL ? L_STG + 4096 : (int)stg) + (lane & 15) * (FSTR == 4 ? 4 : 16);
  const unsigned vS2 = (unsigned)(wc * 512 + (lane & 31) * 16) + (unsigned)(lane >> 5) * KSTEP;
  const unsigned vS1 = lane < 32 ? vS2 : OOB;
  const unsigned vF = (unsigned)rank * FSRC + (unsigned)unit * FSTR;

#define SA(b, h) (((b) * 2 + (h)) * HTB)
#define SB(b, h) ((4 + (b) * 2 + (h)) * HTB)
#define PA_(b, h, kt, i) asm volatile("s_mov_b32 m0, %0\n\ts_nop 0\n\tbuffer_load_dwordx4 %1, %2, %3 offen offset:" #i " sc1 lds" \
    :: "s"(ldq + SA(b, h)), "v"(vA), "s"(srdA), "s"(sA0 + (unsigned)(kt) * KSTEP + (h) * HTB) : "m0", "memory")
#define PB_(b, h, kt, v, i) asm volatile("s_mov_b32 m0, %0\n\ts_nop 0\n\tbuffer_load_dwordx4 %1, %2, %3 offen offset:" #i " lds" \
    :: "s"(ldq + SB(b, h)), "v"(v), "s"(srdB), "s"(sB0 + (unsigned)(kt) * KSTEP + (h) * 256) : "m0", "memory")
#define PAN(b, h, kt, i) asm volatile("buffer_load_dwordx4 %0, %1, %2 offen offset:" #i " sc1 lds" \
    :: "v"(vA), "s"(srdA), "s"(sA0 + (unsigned)(kt) * KSTEP + (h) * HTB) : "memory")
#define PBN(b, h, kt, v, i) asm volatile("buffer_load_dwordx4 %0, %1, %2 offen offset:" #i " lds" \
    :: "v"(v), "s"(srdB), "s"(sB0 + (unsigned)(kt) * KSTEP + (h) * 256) : "memory")
#define STAGE_A(b, h, kt) do { PA_(b, h, kt, 0); PAN(b, h, kt, 1024); PAN(b, h, kt, 2048); PAN(b, h, kt, 3072); } while (0)
#define STAGE_B(b, h, kt) do { PB_(b, h, kt, vB0, 0); PBN(b, h, kt, vB1, 1024); PBN(b, h, kt, vB2, 2048); PBN(b, h, kt, vB3, 3072); } while (0)
#define TR(off) __builtin_amdgcn_ds_read_tr16_b64_v4i16((LAS s16x4*)(lds + (off)))
#define LDA2(dst, b, h, m0_) do { _Pragma("unroll") for (int m = m0_; m < m0_ + 2; ++m) _Pragma("unroll") for (int k = 0; k < 2; ++k) dst[m][k] = *(const LAS bf16x8*)(lds + SA(b, h) + aoff + m * 2048 + k * 1024); } while (0)
#define LDB1(dst, b, h, k) do { \
    { s16x4 lo = TR(boff0 + ((b) * 2 + (h)) * HTB + k * 8192), hi = TR(boff0 + ((b) * 2 + (h)) * HTB + k * 8192 + 1024); dst[0][k] = __builtin_shufflevector(lo, hi, 0, 1, 2, 3, 4, 5, 6, 7); } \
    { s16x4 lo = TR(boff1 + ((b) * 2 + (h)) * HTB + k * 8192), hi = TR(boff1 + ((b) * 2 + (h)) * HTB + k * 8192 + 1024); dst[1][k] = __builtin_shufflevector(lo, hi, 0, 1, 2, 3, 4, 5, 6, 7); } } while (0)
#define WAIT_V(n) asm volatile("s_waitcnt vmcnt(" #n ")" ::: "memory")
#define WAIT_L(n) do { asm volatile("" ::: "memory"); __builtin_amdgcn_s_waitcnt(0xC07F | ((n) << 8)); asm volatile("" ::: "memory"); } while (0)
#define BAR __builtin_amdgcn_s_barrier()
#define SCHED __builtin_amdgcn_sched_barrier(0)
#define MMA(ai, bj, At, Bt) do { _Pragma("unroll") for (int n = 0; n < 2; ++n) _Pragma("unroll") for (int m_ = 0; m_ < 4; ++m_) _Pragma("unroll") for (int k_ = 0; k_ < 2; ++k_) { \
    const int m = n ? 3 - m_ : m_, k = k_ ^ (m_ & 1); \
    acc[ai][bj][m][n] = __builtin_amdgcn_mfma_f32_16x16x32_bf16(Bt[n][k], At[m][k], acc[ai][bj][m][n], 0, 0, 0); SCHED; } } while (0)
#define MMA2(ai, At, B0, B1) do { SCHED; MMA(ai, 0, At, B0); MMA(ai, 1, At, B1); SCHED; } while (0)
#define STAMP(i) do {} while (0)
#define STAMPC(i) do {} while (0)
#define STAMPT(t) do {} while (0)
#define SLOWCNT do {} while (0)
#define PUSH_RD(sl) (*(const LAS u32x4*)(lds + stgl + (sl) * 8192))
#define PUBLISH(val, voff, soff) do { __builtin_amdgcn_raw_buffer_store_b32(base + (unsigned)(val), srdFd, voff, soff, 17); asm volatile("" ::: "memory"); } while (0)
#define CHECK_ON true
#define CHECK(fv, kp) do { if (__builtin_expect(CHECK_ON && (kp) < NKT && !__all((int)((fv) - (base + (unsigned)(kp) + 1u)) >= 0), 0)) { \
        SLOWCNT; SPIN(kp); } } while (0)
#define SPIN(kp) do { if (wid == 4) { spin_flags(myflags + ((kp) & 7) * (FSLOT / 4), base + (unsigned)(kp) + 1u, tmo, call, dead); \
          if (lane == 0) *(volatile LAS unsigned*)(dead + 1) = (unsigned)(kp) + 1u; } \
        else while (__builtin_amdgcn_readfirstlane(*(volatile LAS unsigned*)(dead + 1)) < (unsigned)(kp) + 1u) __builtin_amdgcn_s_sleep(1); \
        asm volatile("" ::: "memory"); } while (0)
#define PACE_UNTIL(ticks) do { while (__builtin_amdgcn_s_memrealtime() - t_start < (unsigned long long)(ticks)) __builtin_amdgcn_s_sleep(1); } while (0)

#define EPI(ai) do { int tz = threadIdx.x; asm volatile("" : "+v"(tz)); \
    const int fr2 = tz & 15, fq2 = (tz >> 4) & 3; const bool odd = fq2 & 1; \
    const __amdgpu_buffer_rsrc_t srdO = mk_rsrc(a.out, (unsigned)MT * NL * 2); \
    const unsigned ooff = ((unsigned)(pm * BM + wr * 64 + fr2) * NL + (unsigned)(pn * BM + wc * 32 + 8 * fq2)) * 2u; \
    if (__builtin_amdgcn_readfirstlane(*dead) != 0u) { \
      _Pragma("unroll") for (int j = 0; j < 2; ++j) _Pragma("unroll") for (int m = 0; m < 4; ++m) _Pragma("unroll") for (int n = 0; n < 2; ++n) \
        acc[ai][j][m][n] = (f32x4){__builtin_nanf(""), __builtin_nanf(""), __builtin_nanf(""), __builtin_nanf("")}; } \
    _Pragma("unroll") for (int m = 0; m < 4; ++m) _Pragma("unroll") for (int bj = 0; bj < 2; ++bj) { \
          const f32x4 v0 = acc[ai][bj][m][0], v1 = acc[ai][bj][m][1]; \
          const unsigned a0 = gelu2((f32x2){v0[0], v0[1]}), a1 = gelu2((f32x2){v0[2], v0[3]}); \
          const unsigned b0 = gelu2((f32x2){v1[0], v1[1]}), b1 = gelu2((f32x2){v1[2], v1[3]}); \
          u32x4 w; w.x = odd ? b0 : a0; w.y = odd ? b1 : a1; w.z = odd ? a0 : b0; w.w = odd ? a1 : b1; \
          st128<16>(w, srdO, ooff + bj * (HALF * 2), (unsigned)((ai) * HALF + m * 16) * (NL * 2)); } } while (0)
  f32x4 acc[2][2][4][2];
#pragma unroll
  for (int i = 0; i < 2; ++i)
#pragma unroll
    for (int j = 0; j < 2; ++j)
#pragma unroll
      for (int m = 0; m < 4; ++m)
#pragma unroll
        for (int n = 0; n < 2; ++n) acc[i][j][m][n] = (f32x4){0.f, 0.f, 0.f, 0.f};
  bf16x8 At[4][2], B0a[2][2], B0b[2][2], B1[2][2];
  if (!wr) {
    if (wc < NPRO) {
      WAIT_V(3);
      const u32x4 pv0 = PREAD(0), pv1 = PREAD(1);
      PACE_UNTIL(wc * PACE);
      st128<17>(pv0, srdD, (unsigned)lane * 16u, sD0 + (unsigned)wc * KSTEP);
      st128<17>(pv1, srdD, (unsigned)lane * 16u + 1024u, sD0 + (unsigned)wc * KSTEP);
      WAIT_V(0);
      PUBLISH(wc + 1, lane == 0 ? vF : OOB, (unsigned)wc * FSLOT);
      if (wc + 4 < NPRO) {
        PLOAD(vs0, wc + 4, 0); PLOAD(vs0 + 64u, wc + 4, 1);
        WAIT_V(0);
        const u32x4 pw0 = PREAD(0), pw1 = PREAD(1);
        PACE_UNTIL((wc + 4) * PACE);
        st128<17>(pw0, srdD, (unsigned)lane * 16u, sD0 + (unsigned)(wc + 4) * KSTEP);
        st128<17>(pw1, srdD, (unsigned)lane * 16u + 1024u, sD0 + (unsigned)(wc + 4) * KSTEP);
        WAIT_V(0);
        PUBLISH(wc + 5, lane == 0 ? vF : OOB, (unsigned)(wc + 4) * FSLOT);
      }
    } else WAIT_V(0);
    STAMP(1);
    { const u32x4 pv2 = PREAD(2);
      PACE_UNTIL(NPRO * PACE);
      st128<17>(pv2, srdD, vS1, sD0 + (unsigned)(LEAD - 1) * KSTEP); }
    WAIT_V(0);
    WAIT_L(0);
    STAMP(2);
    BAR;
    LDB1(B0a, 0, 0, 0); LDB1(B0a, 0, 0, 1);
    WAIT_L(0);
    BAR;
    STAMP(5); STAMPC(44);
    unsigned vF1 = tid == 0 ? vF : OOB, vF2 = tid < 2 ? vF : OOB;
    asm volatile("" : "+v"(vF1), "+v"(vF2));
    u32x4 pv = PUSH_RD(0);
    unsigned fval, foff, soff;
#define PUB_PREP(t) do { const int kf_ = KP((t) - PDEL); fval = base + (unsigned)(kf_ + 1 + lane); \
      foff = ((KP((t) - PDEL + 1) - kf_ == 2 && kf_ + 1 < NKT) ? vF2 : vF1) + (unsigned)((kf_ + lane) & 7) * FSLOT + (kf_ < NKT ? 0u : OOB); \
      soff = KP((t) + 1) - KP(t) == 2 ? vS2 : vS1; asm volatile("" : "+v"(fval), "+v"(foff), "+v"(soff)); } while (0)
    PUB_PREP(0);
#define PUBLAST 0
#define WAIT_MB WAIT_V(9)
#define KTILE0(t, b, B0c, B0n) do { \
    if (!PUBLAST) { __builtin_amdgcn_raw_buffer_store_b32(fval, srdFd, foff, 0u, 17); asm volatile("" ::: "memory"); } \
    st128<17>(pv, srdD, soff, sD0 + (unsigned)KP(t) * KSTEP); \
    if (PUBLAST) { __builtin_amdgcn_raw_buffer_store_b32(fval, srdFd, foff, 0u, 17); asm volatile("" ::: "memory"); } \
    LDB1(B1, b, 1, 0); PB_(b, 0, (t) + 2, vB0, 0); LDB1(B1, b, 1, 1); PBN(b, 0, (t) + 2, vB1, 1024); \
    LDA2(At, b, 0, 0); PBN(b, 0, (t) + 2, vB2, 2048); LDA2(At, b, 0, 2); PBN(b, 0, (t) + 2, vB3, 3072); \
    WAIT_L(0); BAR; MMA2(0, At, B0c, B1); WAIT_V(10); BAR; SCHED; \
    XLOAD(KP((t) + 2), b); pv = PUSH_RD((b) ^ 1); \
    LDA2(At, b, 1, 0); PB_(b, 1, (t) + 2, vB0, 0); LDA2(At, b, 1, 2); PBN(b, 1, (t) + 2, vB1, 1024); \
    PUB_PREP((t) + 1); \
    LDB1(B0n, (b) ^ 1, 0, 0); PBN(b, 1, (t) + 2, vB2, 2048); LDB1(B0n, (b) ^ 1, 0, 1); PBN(b, 1, (t) + 2, vB3, 3072); \
    WAIT_L(0); BAR; MMA2(1, At, B0c, B1); WAIT_MB; BAR; SCHED; } while (0)
    for (int t = 0; t < NKT - 2 * TAIL; t += 2) {
      KTILE0(t, 0, B0a, B0b);
      KTILE0(t + 1, 1, B0b, B0a);
      STAMPT(t);
    }
    BAR;
    STAMPC(45);
  } else {
    PACE_UNTIL(BDELAY);
    STAGE_B(0, 0, 0); STAGE_B(0, 1, 0); STAGE_B(1, 0, 1); STAGE_B(1, 1, 1);
    STAMP(1);
    spin_flags(myflags, base + 1u, tmo, call, dead);
    STAMP(2);
    STAGE_A(0, 0, 0); STAGE_A(0, 1, 0);
    POLL(1);
    WAIT_V(0);
    STAMP(4);
    BAR;
    LDB1(B0a, 0, 0, 0); LDB1(B0a, 0, 0, 1);
    WAIT_L(0);
    BAR;
    STAMP(5); STAMPC(44);
    BAR;
#define KTILE1(t, b, B0c, B0n) do { \
    { const unsigned fv_ = *(const LAS unsigned*)(lds + chkl); asm volatile("" ::: "memory"); \
      LDB1(B1, b, 1, 0); \
      CHECK(fv_, (t) + 1); } \
    PA_((b) ^ 1, 0, (t) + 1, 0); LDB1(B1, b, 1, 1); PAN((b) ^ 1, 0, (t) + 1, 1024); LDA2(At, b, 0, 0); \
    PAN((b) ^ 1, 0, (t) + 1, 2048); LDA2(At, b, 0, 2); PAN((b) ^ 1, 0, (t) + 1, 3072); \
    WAIT_V(4); WAIT_L(0); BAR; MMA2(0, At, B0c, B1); BAR; SCHED; \
    POLL((t) + 2); \
    PA_((b) ^ 1, 1, (t) + 1, 0); LDA2(At, b, 1, 0); PAN((b) ^ 1, 1, (t) + 1, 1024); LDA2(At, b, 1, 2); \
    PAN((b) ^ 1, 1, (t) + 1, 2048); LDB1(B0n, (b) ^ 1, 0, 0); PAN((b) ^ 1, 1, (t) + 1, 3072); LDB1(B0n, (b) ^ 1, 0, 1); \
    WAIT_V(5); WAIT_L(0); BAR; MMA2(1, At, B0c, B1); WAIT_V(4); BAR; SCHED; } while (0)
    for (int t = 0; t < NKT - 2 * TAIL; t += 2) {
      KTILE1(t, 0, B0a, B0b);
      KTILE1(t + 1, 1, B0b, B0a);
      STAMPT(t);
    }
    STAMPC(45);
  }
  EPI(0);
  EPI(1);
  STAMP(40);
  if (tid == 0) __hip_atomic_store((unsigned*)(ws + WS_CTR + (size_t)bid * 128), call, __ATOMIC_RELAXED, __HIP_MEMORY_SCOPE_AGENT);
}
}

extern "C" void kernel_launch(int rank, int nranks, ncclComm_t comm,
                              hipStream_t stream,
                              void* const* d_in, const int* in_sizes, int n_in,
                              void* d_out, int out_size,
                              const int* in_dtypes, int out_dtype,
                              void* const* d_out_all,
                              void* const* d_ws_all, size_t ws_bytes) {
  using namespace agg;
  if (nranks != NR || (unsigned)rank >= (unsigned)NR || n_in < 2 || ws_bytes < WS_NEED || in_sizes[0] != MS * KT || in_sizes[1] != KT * NL || out_size != MT * NL) {
    fprintf(stderr, "kernel_launch: unexpected problem shape\n");
    return;
  }
  static unsigned attr_done = 0;
  int dev = 0; HIP_CHECK(hipGetDevice(&dev));
  if (!(attr_done >> (dev & 31) & 1u)) {
    HIP_CHECK(hipFuncSetAttribute((const void*)ag_gemm_kernel, hipFuncAttributeMaxDynamicSharedMemorySize, LDS_BYTES));
    attr_done |= 1u << (dev & 31);
  }
  Args a;
  a.x = (const char*)d_in[0]; a.W = (const char*)d_in[1]; a.out = (char*)d_out; a.rank = rank;
  for (int p = 0; p < NR; ++p) a.ws[p] = (char*)d_ws_all[(rank + p) & (NR - 1)];
  hipLaunchKernelGGL(ag_gemm_kernel, dim3(256), dim3(512), LDS_BYTES, stream, a);
  HIP_CHECK(hipGetLastError());
}
```

```cpp
#include <hip/hip_runtime.h>
#include <rccl/rccl.h>
#include <cstdio>

#define HIP_CHECK(expr) do { hipError_t _e = (expr); if (_e != hipSuccess) { \
    fprintf(stderr, "HIP error %s at %s:%d\n", hipGetErrorString(_e), __FILE__, __LINE__); \
    return; } } while (0)

namespace agg {
#define LAS __attribute__((address_space(3)))
typedef short bf16x8 __attribute__((ext_vector_type(8)));
typedef short s16x4 __attribute__((ext_vector_type(4)));
typedef float f32x4 __attribute__((ext_vector_type(4)));
typedef float f32x2 __attribute__((ext_vector_type(2)));
typedef unsigned u32x4 __attribute__((ext_vector_type(4)));

constexpr int NR = 8;
constexpr int MT = 4096, KT = 4096, NL = 4096;
constexpr int MS = MT / NR;
constexpr int BM = 256, BK = 64, HALF = 128, HTB = 16384, NKT = KT / BK;
constexpr unsigned KSTEP = 32u * HTB;
static_assert(KSTEP == 64u * NL * 2u, "");
constexpr unsigned IMG_BYTES = (unsigned)MT * KT * 2;
constexpr unsigned X_BYTES = (unsigned)MS * KT * 2, W_BYTES = (unsigned)KT * NL * 2, O_BYTES = (unsigned)MT * NL * 2;
constexpr size_t WS_CTR = 0;
constexpr size_t WS_TMO = 32768;
#define FSTR 16
constexpr size_t WS_FLAG = 65536;
constexpr unsigned FSLOT = 32 * FSTR, FSRC = 8 * FSLOT, FLAG_BYTES = NR * FSRC;
static_assert(FSTR >= 16 && WS_FLAG + FLAG_BYTES <= 524288, "");
constexpr size_t WS_IMG = 1u << 20;
constexpr size_t WS_NEED = WS_IMG + 2 * (size_t)IMG_BYTES;
constexpr int L_STG = 8 * HTB;
constexpr int L_POLL = L_STG + 8192;
constexpr int L_DEAD = L_POLL + 4096;
constexpr int L_PRO = L_STG + 16384;
constexpr int LDS_BYTES = L_PRO + 8192;
static_assert(LDS_BYTES <= 160 * 1024, "");
constexpr unsigned OOB = 0x40000000u;
#define LEAD 4
#define WPOS 2
#define PACE 110
#define BDELAY 40
constexpr unsigned long long DEADLINE = 100000000ull;

struct Args { const char* x; const char* W; char* out; char* ws[NR]; int rank; };

__host__ __device__ __forceinline__ int lds_byte(int r, int c) { const int st = (r >> 4) * 2 + (c >> 5), rr = r & 15, cc = c & 31, ob = rr * 64 + cc * 2; return st * 1024 + (ob ^ (((ob >> 9) & 1) << 5)); }
__host__ __device__ __forceinline__ void stage_rc(int b, int& R, int& C) { const int st = b / 1024, sb = b % 1024, swz = sb ^ (((sb >> 9) & 1) << 5); R = (st >> 1) * 16 + swz / 64; C = (st & 1) * 32 + (swz % 64) / 2; }
__host__ __device__ __forceinline__ int fk(int k) { return (k & 1) | ((k & 2) << 1) | (k & 8); }

__device__ __forceinline__ unsigned cvt_pk_bf16(float lo, float hi) { unsigned r; asm("v_cvt_pk_bf16_f32 %0, %1, %2" : "=v"(r) : "v"(lo), "v"(hi)); return r; }
__device__ __forceinline__ unsigned gelu2(f32x2 y) {
  const float k1 = -2.0f * 0.7978845608028654f * 1.4426950408889634f, k2 = k1 * 0.044715f;
  const f32x2 z = y * __builtin_elementwise_fma(y * y, (f32x2){k2, k2}, (f32x2){k1, k1});
  f32x2 e; e.x = __builtin_amdgcn_exp2f(z.x); e.y = __builtin_amdgcn_exp2f(z.y);
  const f32x2 d = e + 1.0f;
  f32x2 r; r.x = __builtin_amdgcn_rcpf(d.x); r.y = __builtin_amdgcn_rcpf(d.y);
  const f32x2 o = y * r;
  return cvt_pk_bf16(o.x, o.y);
}
template <int AUX> __device__ __forceinline__ void st128(u32x4 v, __amdgpu_buffer_rsrc_t r, unsigned voff, unsigned soff) {
  __builtin_amdgcn_raw_buffer_store_b128(v, r, voff, soff, AUX);
  asm volatile("s_nop 1" :: "v"(v) : "memory");
}
__device__ __forceinline__ __amdgpu_buffer_rsrc_t mk_rsrc(const void* p, unsigned bytes) { return __builtin_amdgcn_make_buffer_rsrc((void*)p, (short)0, (int)bytes, 0x00020000); }

__device__ __forceinline__ void spin_flags(const unsigned* f, unsigned need, unsigned* tmo, unsigned call, LAS unsigned* dead) {
  const unsigned long long t0 = __builtin_amdgcn_s_memrealtime();
  __builtin_amdgcn_s_waitcnt(0xC07F);
  for (;;) {
    const unsigned v = __hip_atomic_load(f, __ATOMIC_RELAXED, __HIP_MEMORY_SCOPE_SYSTEM);
    if (__all((int)(v - need) >= 0)) break;
    if (__builtin_amdgcn_readfirstlane(__hip_atomic_load(tmo, __ATOMIC_RELAXED, __HIP_MEMORY_SCOPE_AGENT)) == call ||
        __builtin_amdgcn_s_memrealtime() - t0 > DEADLINE) {
      __hip_atomic_store(tmo, call, __ATOMIC_RELAXED, __HIP_MEMORY_SCOPE_AGENT);
      *dead = 1u;
      asm volatile("s_waitcnt vmcnt(0) lgkmcnt(0)" ::: "memory");
      break;
    }
    __builtin_amdgcn_s_sleep(1);
  }
  asm volatile("" ::: "memory");
}

__global__ __launch_bounds__(512, 2) void ag_gemm_kernel(const Args a) {
  extern __shared__ __attribute__((aligned(16))) unsigned char shm_[];
  LAS unsigned char* lds = (LAS unsigned char*)shm_;
  const int tid = threadIdx.x, wid = __builtin_amdgcn_readfirstlane(tid >> 6), lane = tid & 63, wr = wid >> 2, wc = wid & 3, fr = lane & 15, fq = lane >> 4;
  const int bid = blockIdx.x, rank = a.rank;
  char* const ws = a.ws[0];
  const int xcd = bid & 7, idx = bid >> 3;
  const int pm = (xcd & 1) * 8 + (idx & 7), pn = (xcd >> 1) * 4 + (idx >> 3);
  const int unit = (bid & 7) | ((bid >> 6) << 3);
  char* const wsd = a.ws[(bid >> 3) & (NR - 1)];
  const __amdgpu_buffer_rsrc_t srdB = mk_rsrc(a.W, W_BYTES);
  const __amdgpu_buffer_rsrc_t srdX = mk_rsrc(a.x, X_BYTES);
  const __amdgpu_buffer_rsrc_t srdFd = mk_rsrc(wsd + WS_FLAG, FLAG_BYTES);
  const __amdgpu_buffer_rsrc_t srdFl = mk_rsrc(ws + WS_FLAG, FLAG_BYTES);
  const unsigned ldsb = (unsigned)(size_t)lds;
  unsigned xs0, vx1;
  { int R, C; stage_rc((unit & 7) * 2048 + lane * 16, R, C); xs0 = (unsigned)(((unit >> 3) * 128 + R) * KT + C) * 2u;
    stage_rc((unit & 7) * 2048 + wc * 512 + (lane & 31) * 16, R, C);
    vx1 = !wr ? (lane < 32 ? (unsigned)(((unit >> 3) * 128 + R) * KT + C) * 2u : OOB)
              : (wid == 4 && lane < 16 ? (unsigned)(pm >> 1) * FSRC + (unsigned)((pm & 1) * 16 + lane) * FSTR : OOB); }
#define XLOAD(kx, sl) asm volatile("s_mov_b32 m0, %0\n\ts_nop 0\n\tbuffer_load_dwordx4 %1, %2, %3 offen lds" \
        :: "s"(ldsb + L_STG + (sl) * 4096 + wc * 1024), "v"(vx1), "s"(srdX), "s"((unsigned)(kx) << 7) : "m0", "memory")
#define POLL(kp) asm volatile("s_mov_b32 m0, %0\n\ts_nop 0\n\tbuffer_load_dwordx4 %1, %2, %3 offen sc0 sc1 lds" \
        :: "s"(ldsb + L_POLL + wc * 1024), "v"(vx1), "s"(srdFl), "s"((unsigned)((kp) & 7) * FSLOT) : "m0", "memory")
#define PLOAD(kx, i) asm volatile("s_mov_b32 m0, %0\n\ts_nop 0\n\tbuffer_load_dwordx4 %1, %2, %3 offen lds" \
        :: "s"(ldsb + L_PRO + wc * 2048 + (i) * 1024), "v"(xs0), "s"(srdX), "s"(((unsigned)(kx) << 7) + (i) * 64u) : "m0", "memory")
#define PREAD(i) (*(const LAS u32x4*)(lds + L_PRO + wc * 2048 + (i) * 1024 + lane * 16))
  LAS unsigned* const dead = (LAS unsigned*)(lds + L_DEAD);
  if (!wr) { PLOAD(wc, 0); PLOAD(wc, 1); XLOAD(LEAD, 0); XLOAD(LEAD + 1, 1); }
  else if (tid == 256) { dead[0] = 0u; dead[1] = 0u; }
  unsigned prev;
  asm volatile("s_load_dword %0, %1, 0x0\n\ts_waitcnt lgkmcnt(0)" : "=s"(prev) : "s"(ws + WS_CTR + (size_t)bid * 128) : "memory");
  const unsigned call = prev + 1u, base = prev * (unsigned)NKT, par = call & 1u;
  const unsigned long long t_start = __builtin_amdgcn_s_memrealtime();
  const __amdgpu_buffer_rsrc_t srdA = mk_rsrc(ws + WS_IMG + (size_t)par * IMG_BYTES, IMG_BYTES);
  const __amdgpu_buffer_rsrc_t srdD = mk_rsrc(wsd + WS_IMG + (size_t)par * IMG_BYTES, IMG_BYTES);
  unsigned* const tmo = (unsigned*)(ws + WS_TMO);
  const unsigned* const myflags = (const unsigned*)(ws + WS_FLAG + (unsigned)(pm >> 1) * FSRC + (unsigned)((pm & 1) * 16 + (lane & 15)) * FSTR);

  const unsigned vA = (unsigned)(wc * 4096 + lane * 16);
  const unsigned vB0 = (unsigned)(wc * 16 + (lane >> 4)) * (NL * 2) + (unsigned)(((lane & 15) ^ fk(lane >> 4)) * 16);
  const unsigned vB1 = vB0 + 4 * (NL * 2) - 1024, vB2 = (vB0 ^ 128u) + 8 * (NL * 2) - 2048, vB3 = (vB0 ^ 128u) + 12 * (NL * 2) - 3072;
  const unsigned ldq = ldsb + (unsigned)wc * 4096u;
  const int aoff = lds_byte(wr * 64 + fr, fq * 8);
  const int q_ = (lane & 15) >> 2, p_ = lane & 3;
  const int f_ = (q_ & 1) | ((q_ & 2) << 1) | ((fq & 1) << 3);
  int boff0 = 4 * HTB + 256 * (8 * fq + q_) + 16 * ((4 * wc + p_) ^ f_) + 8 * (p_ & 1);
  int boff1 = boff0 ^ 8;
  asm volatile("" : "+v"(boff0), "+v"(boff1));
  const unsigned sA0 = (unsigned)(2 * pm) * HTB, sB0 = (unsigned)pn * 512u;
  const unsigned sD0 = (unsigned)(rank * 4 + (unit >> 3)) * HTB + (unsigned)(unit & 7) * 2048u;
  const unsigned vF = (unsigned)rank * FSRC + (unsigned)unit * FSTR;

#define SA(b, h) (((b) * 2 + (h)) * HTB)
#define SB(b, h) ((4 + (b) * 2 + (h)) * HTB)
#define PA_(b, h, kt) asm volatile("s_mov_b32 m0, %0\n\ts_nop 0\n\tbuffer_load_dwordx4 %1, %2, %3 offen sc0 sc1 lds" \
    :: "s"(ldq + SA(b, h)), "v"(vA), "s"(srdA), "s"(sA0 + (unsigned)(kt) * KSTEP + (h) * HTB) : "m0", "memory")
#define PB_(b, h, kt) asm volatile("s_mov_b32 m0, %0\n\ts_nop 0\n\tbuffer_load_dwordx4 %1, %2, %3 offen lds" \
    :: "s"(ldq + SB(b, h)), "v"(vB0), "s"(srdB), "s"(sB0 + (unsigned)(kt) * KSTEP + (h) * 256) : "m0", "memory")
#define PAN(h, kt, i) asm volatile("buffer_load_dwordx4 %0, %1, %2 offen offset:" #i " sc0 sc1 lds" \
    :: "v"(vA), "s"(srdA), "s"(sA0 + (unsigned)(kt) * KSTEP + (h) * HTB) : "memory")
#define PBN(h, kt, v, i) asm volatile("buffer_load_dwordx4 %0, %1, %2 offen offset:" #i " lds" \
    :: "v"(v), "s"(srdB), "s"(sB0 + (unsigned)(kt) * KSTEP + (h) * 256) : "memory")
#define STAGE_A(b, h, kt) do { PA_(b, h, kt); PAN(h, kt, 1024); PAN(h, kt, 2048); PAN(h, kt, 3072); } while (0)
#define STAGE_B(b, h, kt) do { PB_(b, h, kt); PBN(h, kt, vB1, 1024); PBN(h, kt, vB2, 2048); PBN(h, kt, vB3, 3072); } while (0)
#define TR(off) __builtin_amdgcn_ds_read_tr16_b64_v4i16((LAS s16x4*)(lds + (off)))
#define LDA2(dst, b, h, m0_) do { _Pragma("unroll") for (int m = m0_; m < m0_ + 2; ++m) _Pragma("unroll") for (int k = 0; k < 2; ++k) dst[m][k] = *(const LAS bf16x8*)(lds + SA(b, h) + aoff + m * 2048 + k * 1024); } while (0)
#define LDB1(dst, b, h, k) do { \
    { s16x4 lo = TR(boff0 + ((b) * 2 + (h)) * HTB + k * 8192), hi = TR(boff0 + ((b) * 2 + (h)) * HTB + k * 8192 + 1024); dst[0][k] = __builtin_shufflevector(lo, hi, 0, 1, 2, 3, 4, 5, 6, 7); } \
    { s16x4 lo = TR(boff1 + ((b) * 2 + (h)) * HTB + k * 8192), hi = TR(boff1 + ((b) * 2 + (h)) * HTB + k * 8192 + 1024); dst[1][k] = __builtin_shufflevector(lo, hi, 0, 1, 2, 3, 4, 5, 6, 7); } } while (0)
#define WAIT_V(n) asm volatile("s_waitcnt vmcnt(" #n ")" ::: "memory")
#define WAIT_L(n) do { asm volatile("" ::: "memory"); __builtin_amdgcn_s_waitcnt(0xC07F | ((n) << 8)); asm volatile("" ::: "memory"); } while (0)
#define BAR do { asm volatile("" ::: "memory"); __builtin_amdgcn_s_barrier(); asm volatile("" ::: "memory"); } while (0)
#define SCHED __builtin_amdgcn_sched_barrier(0)
#define MMA(ai, bj, At, Bt) do { _Pragma("unroll") for (int n = 0; n < 2; ++n) _Pragma("unroll") for (int m_ = 0; m_ < 4; ++m_) _Pragma("unroll") for (int k_ = 0; k_ < 2; ++k_) { \
    const int m = n ? 3 - m_ : m_, k = k_ ^ (m_ & 1); \
    acc[ai][bj][m][n] = __builtin_amdgcn_mfma_f32_16x16x32_bf16(Bt[n][k], At[m][k], acc[ai][bj][m][n], 0, 0, 0); SCHED; } } while (0)
#define MMA2(ai, At, B0, B1) do { SCHED; MMA(ai, 0, At, B0); MMA(ai, 1, At, B1); SCHED; } while (0)
#define MMA2W(ai, At, B0, B1, W) do { SCHED; MMA(ai, 0, At, B0); \
    _Pragma("unroll") for (int n = 0; n < 2; ++n) _Pragma("unroll") for (int m_ = 0; m_ < 4; ++m_) { if (n == 1 && m_ == WPOS) { W; SCHED; } _Pragma("unroll") for (int k_ = 0; k_ < 2; ++k_) { \
    const int m = n ? 3 - m_ : m_, k = k_ ^ (m_ & 1); \
    acc[ai][1][m][n] = __builtin_amdgcn_mfma_f32_16x16x32_bf16(B1[n][k], At[m][k], acc[ai][1][m][n], 0, 0, 0); SCHED; } } SCHED; } while (0)
#define PUBLISH(k, voff) do { __builtin_amdgcn_raw_buffer_store_b32(base + (unsigned)(k) + 1u, srdFd, voff, (k) < NKT ? (unsigned)((k) & 7) * FSLOT : OOB, 17); asm volatile("" ::: "memory"); } while (0)
#define PACE_UNTIL(ticks) do { while (__builtin_amdgcn_s_memrealtime() - t_start < (unsigned long long)(ticks)) __builtin_amdgcn_s_sleep(1); } while (0)

  f32x4 acc[2][2][4][2];
#pragma unroll
  for (int i = 0; i < 2; ++i)
#pragma unroll
    for (int j = 0; j < 2; ++j)
#pragma unroll
      for (int m = 0; m < 4; ++m)
#pragma unroll
        for (int n = 0; n < 2; ++n) acc[i][j][m][n] = (f32x4){0.f, 0.f, 0.f, 0.f};
  bf16x8 At[4][2], B0a[2][2], B0b[2][2], B1[2][2];
  if (!wr) {
#pragma unroll
    for (int k0 = 0; k0 < LEAD; k0 += 4) {
      const int k = k0 + wc;
      if (k < LEAD) {
        if (k0) { PLOAD(k, 0); PLOAD(k, 1); WAIT_V(0); } else WAIT_V(2);
        const u32x4 p0 = PREAD(0), p1 = PREAD(1);
        PACE_UNTIL(k * PACE);
        st128<17>(p0, srdD, (unsigned)lane * 16u, sD0 + (unsigned)k * KSTEP);
        st128<17>(p1, srdD, (unsigned)lane * 16u + 1024u, sD0 + (unsigned)k * KSTEP);
        WAIT_V(0);
        PUBLISH(k, lane == 0 ? vF : OOB);
      }
    }
    if (LEAD < 4) WAIT_V(0);
    WAIT_L(0);
    BAR;
    LDB1(B0a, 0, 0, 0); LDB1(B0a, 0, 0, 1);
    WAIT_L(0);
    BAR;
    unsigned vS = lane < 32 ? (unsigned)(wc * 512 + lane * 16) : OOB, vF0 = tid == 0 ? vF : OOB;
    asm volatile("" : "+v"(vS), "+v"(vF0));
    const int stgl = L_STG + wc * 1024 + lane * 16;
#define PUSH_RD(sl) (*(const LAS u32x4*)(lds + stgl + (sl) * 4096))
    u32x4 pv = PUSH_RD(0);
#define KTILE0(t, b, B0c, B0n) do { \
    PUBLISH((t) + LEAD - 1, vF0); \
    st128<17>(pv, srdD, vS, sD0 + (unsigned)((t) + LEAD) * KSTEP); \
    LDB1(B1, b, 1, 0); PB_(b, 0, (t) + 2); LDB1(B1, b, 1, 1); PBN(0, (t) + 2, vB1, 1024); \
    LDA2(At, b, 0, 0); PBN(0, (t) + 2, vB2, 2048); LDA2(At, b, 0, 2); PBN(0, (t) + 2, vB3, 3072); \
    WAIT_L(0); BAR; MMA2W(0, At, B0c, B1, WAIT_V(10)); BAR; SCHED; \
    XLOAD((t) + LEAD + 2, b); pv = PUSH_RD((b) ^ 1); \
    LDA2(At, b, 1, 0); PB_(b, 1, (t) + 2); LDA2(At, b, 1, 2); PBN(1, (t) + 2, vB1, 1024); \
    LDB1(B0n, (b) ^ 1, 0, 0); PBN(1, (t) + 2, vB2, 2048); LDB1(B0n, (b) ^ 1, 0, 1); PBN(1, (t) + 2, vB3, 3072); \
    WAIT_L(0); BAR; MMA2W(1, At, B0c, B1, WAIT_V(9)); BAR; SCHED; } while (0)
    for (int t = 0; t < NKT; t += 2) {
      KTILE0(t, 0, B0a, B0b);
      KTILE0(t + 1, 1, B0b, B0a);
    }
    BAR;
  } else {
    PACE_UNTIL(BDELAY);
    STAGE_B(0, 0, 0); STAGE_B(0, 1, 0); STAGE_B(1, 0, 1); STAGE_B(1, 1, 1);
    spin_flags(myflags, base + 1u, tmo, call, dead);
    STAGE_A(0, 0, 0); STAGE_A(0, 1, 0);
    POLL(1);
    WAIT_V(0);
    BAR;
    LDB1(B0a, 0, 0, 0); LDB1(B0a, 0, 0, 1);
    WAIT_L(0);
    BAR;
    BAR;
    const int chkl = L_POLL + (lane & 15) * 16;
#define FLAGS_OK(fv, kp) __all((int)((fv) - (base + (unsigned)(kp) + 1u)) >= 0)
#define CHECK(fv, kp) do { if (__builtin_expect((kp) < NKT && !FLAGS_OK(fv, kp), 0)) { \
        if (wid == 4) { WAIT_V(4); \
          if (!FLAGS_OK(*(volatile LAS unsigned*)(lds + chkl), kp)) { spin_flags(myflags + ((kp) & 7) * (FSLOT / 4), base + (unsigned)(kp) + 1u, tmo, call, dead); \
            if (lane == 0) *(volatile LAS unsigned*)(dead + 1) = (unsigned)(kp) + 1u; } } \
        else while (!FLAGS_OK(*(volatile LAS unsigned*)(lds + chkl), kp) && __builtin_amdgcn_readfirstlane(*(volatile LAS unsigned*)(dead + 1)) < (unsigned)(kp) + 1u) __builtin_amdgcn_s_sleep(1); \
        asm volatile("" ::: "memory"); } } while (0)
#define KTILE1(t, b, B0c, B0n) do { \
    { const unsigned fv_ = *(const LAS unsigned*)(lds + chkl); asm volatile("" ::: "memory"); \
      LDB1(B1, b, 1, 0); \
      CHECK(fv_, (t) + 1); } \
    PA_((b) ^ 1, 0, (t) + 1); LDB1(B1, b, 1, 1); PAN(0, (t) + 1, 1024); LDA2(At, b, 0, 0); \
    PAN(0, (t) + 1, 2048); LDA2(At, b, 0, 2); PAN(0, (t) + 1, 3072); \
    WAIT_V(4); WAIT_L(0); BAR; MMA2(0, At, B0c, B1); BAR; SCHED; \
    POLL((t) + 2); \
    PA_((b) ^ 1, 1, (t) + 1); LDA2(At, b, 1, 0); PAN(1, (t) + 1, 1024); LDA2(At, b, 1, 2); \
    PAN(1, (t) + 1, 2048); LDB1(B0n, (b) ^ 1, 0, 0); PAN(1, (t) + 1, 3072); LDB1(B0n, (b) ^ 1, 0, 1); \
    WAIT_V(5); WAIT_L(0); BAR; MMA2(1, At, B0c, B1); BAR; SCHED; } while (0)
    for (int t = 0; t < NKT; t += 2) {
      KTILE1(t, 0, B0a, B0b);
      KTILE1(t + 1, 1, B0b, B0a);
    }
  }
  {
    int tz = threadIdx.x; asm volatile("" : "+v"(tz));
    const int fr2 = tz & 15, fq2 = (tz >> 4) & 3;
    const bool odd = fq2 & 1;
    const __amdgpu_buffer_rsrc_t srdO = mk_rsrc(a.out, O_BYTES);
    const unsigned ooff = ((unsigned)(pm * BM + wr * 64 + fr2) * NL + (unsigned)(pn * BM + wc * 32 + 8 * fq2)) * 2u;
    if (__builtin_amdgcn_readfirstlane(*dead) != 0u) {
#pragma unroll
      for (int i = 0; i < 2; ++i)
#pragma unroll
        for (int j = 0; j < 2; ++j)
#pragma unroll
          for (int m = 0; m < 4; ++m)
#pragma unroll
            for (int n = 0; n < 2; ++n) acc[i][j][m][n] = (f32x4){__builtin_nanf(""), __builtin_nanf(""), __builtin_nanf(""), __builtin_nanf("")};
    }
#pragma unroll
    for (int ai = 0; ai < 2; ++ai)
#pragma unroll
      for (int m = 0; m < 4; ++m)
#pragma unroll
        for (int bj = 0; bj < 2; ++bj) {
          const f32x4 v0 = acc[ai][bj][m][0], v1 = acc[ai][bj][m][1];
          const unsigned a0 = gelu2((f32x2){v0[0], v0[1]}), a1 = gelu2((f32x2){v0[2], v0[3]});
          const unsigned b0 = gelu2((f32x2){v1[0], v1[1]}), b1 = gelu2((f32x2){v1[2], v1[3]});
          u32x4 w; w.x = odd ? b0 : a0; w.y = odd ? b1 : a1; w.z = odd ? a0 : b0; w.w = odd ? a1 : b1;
          st128<16>(w, srdO, ooff + bj * (HALF * 2), (unsigned)(ai * HALF + m * 16) * (NL * 2));
        }
    if (tz == 0) __hip_atomic_store((unsigned*)(ws + WS_CTR + (size_t)bid * 128), call, __ATOMIC_RELAXED, __HIP_MEMORY_SCOPE_AGENT);
  }
}
}

extern "C" void kernel_launch(int rank, int nranks, ncclComm_t comm,
                              hipStream_t stream,
                              void* const* d_in, const int* in_sizes, int n_in,
                              void* d_out, int out_size,
                              const int* in_dtypes, int out_dtype,
                              void* const* d_out_all,
                              void* const* d_ws_all, size_t ws_bytes) {
  using namespace agg;
  if (nranks != NR || (unsigned)rank >= (unsigned)NR || n_in < 2 || ws_bytes < WS_NEED || in_sizes[0] != MS * KT || in_sizes[1] != KT * NL || out_size != MT * NL) {
    fprintf(stderr, "kernel_launch: unexpected problem shape\n");
    return;
  }
  static unsigned attr_done = 0;
  int dev = 0; HIP_CHECK(hipGetDevice(&dev));
  if (!(attr_done >> (dev & 31) & 1u)) {
    HIP_CHECK(hipFuncSetAttribute((const void*)ag_gemm_kernel, hipFuncAttributeMaxDynamicSharedMemorySize, LDS_BYTES));
    attr_done |= 1u << (dev & 31);
  }
  Args a;
  a.x = (const char*)d_in[0]; a.W = (const char*)d_in[1]; a.out = (char*)d_out; a.rank = rank;
  for (int p = 0; p < NR; ++p) a.ws[p] = (char*)d_ws_all[(rank + p) & (NR - 1)];
  hipLaunchKernelGGL(ag_gemm_kernel, dim3(256), dim3(512), LDS_BYTES, stream, a);
  HIP_CHECK(hipGetLastError());
}
```

```cpp
#include <hip/hip_runtime.h>
#include <rccl/rccl.h>
#include <cstdio>

#define HIP_CHECK(expr) do { hipError_t _e = (expr); if (_e != hipSuccess) { \
    fprintf(stderr, "HIP error %s at %s:%d\n", hipGetErrorString(_e), __FILE__, __LINE__); \
    return; } } while (0)

namespace agg {
#define LAS __attribute__((address_space(3)))
typedef short bf16x8 __attribute__((ext_vector_type(8)));
typedef short s16x4 __attribute__((ext_vector_type(4)));
typedef float f32x4 __attribute__((ext_vector_type(4)));
typedef float f32x2 __attribute__((ext_vector_type(2)));
typedef unsigned u32x4 __attribute__((ext_vector_type(4)));

constexpr int NR = 8;
constexpr int MT = 4096, KT = 4096, NL = 4096;
constexpr int MS = MT / NR;
constexpr int BM = 256, BK = 64, HALF = 128, HTB = 16384, NKT = KT / BK;
constexpr unsigned KSTEP = 64u * NL * 2u;
constexpr unsigned ASTEP = 4u * HTB;
constexpr unsigned IMG_BYTES = (unsigned)MT * KT * 2;
constexpr unsigned X_BYTES = (unsigned)MS * KT * 2, W_BYTES = (unsigned)KT * NL * 2, O_BYTES = (unsigned)MT * NL * 2;
constexpr size_t WS_CTR = 0;
constexpr size_t WS_TMO = 32768;
#define FSTR 16
constexpr size_t WS_FLAG = 65536;
constexpr unsigned FSLOT = 32 * FSTR, FSRC = 8 * FSLOT, FLAG_BYTES = NR * FSRC;
static_assert(FSTR >= 16 && WS_FLAG + FLAG_BYTES <= 524288, "");
constexpr size_t WS_IMG = 1u << 20;
constexpr size_t WS_NEED = WS_IMG + 2 * (size_t)IMG_BYTES;
constexpr int L_STG = 8 * HTB;
constexpr int L_POLL = L_STG + 8192;
constexpr int L_DEAD = L_POLL + 4096;
constexpr int L_PRO = L_STG + 16384;
constexpr int LDS_BYTES = L_PRO + 8192;
static_assert(LDS_BYTES <= 160 * 1024, "");
constexpr unsigned OOB = 0x40000000u;
#define LEAD 4
#define WPOS 2
#define PACE 110
#define BDELAY 40
constexpr unsigned long long DEADLINE = 100000000ull;

struct Args { const char* x; const char* W; char* out; char* ws[NR]; int rank; };

__host__ __device__ __forceinline__ int lds_byte(int r, int c) { const int st = (r >> 4) * 2 + (c >> 5), rr = r & 15, cc = c & 31, ob = rr * 64 + cc * 2; return st * 1024 + (ob ^ (((ob >> 9) & 1) << 5)); }
__host__ __device__ __forceinline__ void stage_rc(int b, int& R, int& C) { const int st = b / 1024, sb = b % 1024, swz = sb ^ (((sb >> 9) & 1) << 5); R = (st >> 1) * 16 + swz / 64; C = (st & 1) * 32 + (swz % 64) / 2; }
__host__ __device__ __forceinline__ int fk(int k) { return (k & 1) | ((k & 2) << 1) | (k & 8); }

__device__ __forceinline__ unsigned cvt_pk_bf16(float lo, float hi) { unsigned r; asm("v_cvt_pk_bf16_f32 %0, %1, %2" : "=v"(r) : "v"(lo), "v"(hi)); return r; }
__device__ __forceinline__ unsigned gelu2(f32x2 y) {
  const float k1 = -2.0f * 0.7978845608028654f * 1.4426950408889634f, k2 = k1 * 0.044715f;
  const f32x2 z = y * __builtin_elementwise_fma(y * y, (f32x2){k2, k2}, (f32x2){k1, k1});
  f32x2 e; e.x = __builtin_amdgcn_exp2f(z.x); e.y = __builtin_amdgcn_exp2f(z.y);
  const f32x2 d = e + 1.0f;
  f32x2 r; r.x = __builtin_amdgcn_rcpf(d.x); r.y = __builtin_amdgcn_rcpf(d.y);
  const f32x2 o = y * r;
  return cvt_pk_bf16(o.x, o.y);
}
template <int AUX> __device__ __forceinline__ void st128(u32x4 v, __amdgpu_buffer_rsrc_t r, unsigned voff, unsigned soff) {
  __builtin_amdgcn_raw_buffer_store_b128(v, r, voff, soff, AUX);
  asm volatile("s_nop 1" :: "v"(v) : "memory");
}
__device__ __forceinline__ __amdgpu_buffer_rsrc_t mk_rsrc(const void* p, unsigned bytes) { return __builtin_amdgcn_make_buffer_rsrc((void*)p, (short)0, (int)bytes, 0x00020000); }

__device__ __forceinline__ void spin_flags(const unsigned* f, unsigned need, unsigned* tmo, unsigned call, LAS unsigned* dead) {
  const unsigned long long t0 = __builtin_amdgcn_s_memrealtime();
  __builtin_amdgcn_s_waitcnt(0xC07F);
  for (;;) {
    const unsigned v = __hip_atomic_load(f, __ATOMIC_RELAXED, __HIP_MEMORY_SCOPE_SYSTEM);
    if (__all((int)(v - need) >= 0)) break;
    if (__builtin_amdgcn_readfirstlane(__hip_atomic_load(tmo, __ATOMIC_RELAXED, __HIP_MEMORY_SCOPE_AGENT)) == call ||
        __builtin_amdgcn_s_memrealtime() - t0 > DEADLINE) {
      __hip_atomic_store(tmo, call, __ATOMIC_RELAXED, __HIP_MEMORY_SCOPE_AGENT);
      *dead = 1u;
      asm volatile("s_waitcnt vmcnt(0) lgkmcnt(0)" ::: "memory");
      break;
    }
    __builtin_amdgcn_s_sleep(1);
  }
  asm volatile("" ::: "memory");
}

__global__ __launch_bounds__(512, 2) void ag_gemm_kernel(const Args a) {
  extern __shared__ __attribute__((aligned(16))) unsigned char shm_[];
  LAS unsigned char* lds = (LAS unsigned char*)shm_;
  const int tid = threadIdx.x, wid = __builtin_amdgcn_readfirstlane(tid >> 6), lane = tid & 63, wr = wid >> 2, wc = wid & 3, fr = lane & 15, fq = lane >> 4;
  const int bid = blockIdx.x, rank = a.rank;
  char* const ws = a.ws[0];
  const int xcd = bid & 7, idx = bid >> 3;
  const int pm = (xcd & 1) * 8 + (idx & 7), pn = (xcd >> 1) * 4 + (idx >> 3);
  const int unit = (bid & 7) | ((bid >> 6) << 3);
  char* const wsd = a.ws[(bid >> 3) & (NR - 1)];
  const __amdgpu_buffer_rsrc_t srdB = mk_rsrc(a.W, W_BYTES);
  const __amdgpu_buffer_rsrc_t srdX = mk_rsrc(a.x, X_BYTES);
  const __amdgpu_buffer_rsrc_t srdFd = mk_rsrc(wsd + WS_FLAG, FLAG_BYTES);
  const __amdgpu_buffer_rsrc_t srdFl = mk_rsrc(ws + WS_FLAG, FLAG_BYTES);
  const unsigned ldsb = (unsigned)(size_t)lds;
  unsigned xs0, vx1;
  { int R, C; stage_rc((unit & 7) * 2048 + lane * 16, R, C); xs0 = (unsigned)(((unit >> 3) * 128 + R) * KT + C) * 2u;
    stage_rc((unit & 7) * 2048 + wc * 512 + (lane & 31) * 16, R, C);
    vx1 = !wr ? (lane < 32 ? (unsigned)(((unit >> 3) * 128 + R) * KT + C) * 2u : OOB)
              : (wid == 4 && lane < 16 ? (unsigned)(pm >> 1) * FSRC + (unsigned)((pm & 1) * 16 + lane) * FSTR : OOB); }
#define XLOAD(kx, sl) asm volatile("s_mov_b32 m0, %0\n\ts_nop 0\n\tbuffer_load_dwordx4 %1, %2, %3 offen lds" \
        :: "s"(ldsb + L_STG + (sl) * 4096 + wc * 1024), "v"(vx1), "s"(srdX), "s"((unsigned)(kx) << 7) : "m0", "memory")
#define POLL(kp) asm volatile("s_mov_b32 m0, %0\n\ts_nop 0\n\tbuffer_load_dwordx4 %1, %2, %3 offen sc0 sc1 lds" \
        :: "s"(ldsb + L_POLL + wc * 1024), "v"(vx1), "s"(srdFl), "s"((unsigned)((kp) & 7) * FSLOT) : "m0", "memory")
#define PLOAD(kx, i) asm volatile("s_mov_b32 m0, %0\n\ts_nop 0\n\tbuffer_load_dwordx4 %1, %2, %3 offen lds" \
        :: "s"(ldsb + L_PRO + wc * 2048 + (i) * 1024), "v"(xs0), "s"(srdX), "s"(((unsigned)(kx) << 7) + (i) * 64u) : "m0", "memory")
#define PREAD(i) (*(const LAS u32x4*)(lds + L_PRO + wc * 2048 + (i) * 1024 + lane * 16))
  LAS unsigned* const dead = (LAS unsigned*)(lds + L_DEAD);
  if (!wr) { PLOAD(wc, 0); PLOAD(wc, 1); XLOAD(LEAD, 0); XLOAD(LEAD + 1, 1); }
  else if (tid == 256) { dead[0] = 0u; dead[1] = 0u; }
  unsigned prev;
  asm volatile("s_load_dword %0, %1, 0x0\n\ts_waitcnt lgkmcnt(0)" : "=s"(prev) : "s"(ws + WS_CTR + (size_t)bid * 128) : "memory");
  const unsigned call = prev + 1u, base = prev * (unsigned)NKT, par = call & 1u;
  const unsigned long long t_start = __builtin_amdgcn_s_memrealtime();
  const __amdgpu_buffer_rsrc_t srdA = mk_rsrc(ws + WS_IMG + (size_t)par * IMG_BYTES + (size_t)(pm >> 1) * X_BYTES, X_BYTES);
  const __amdgpu_buffer_rsrc_t srdD = mk_rsrc(wsd + WS_IMG + (size_t)par * IMG_BYTES + (size_t)rank * X_BYTES, X_BYTES);
  unsigned* const tmo = (unsigned*)(ws + WS_TMO);
  const unsigned* const myflags = (const unsigned*)(ws + WS_FLAG + (unsigned)(pm >> 1) * FSRC + (unsigned)((pm & 1) * 16 + (lane & 15)) * FSTR);

  const unsigned vA = (unsigned)(wc * 4096 + lane * 16);
  const unsigned vB0 = (unsigned)(wc * 16 + (lane >> 4)) * (NL * 2) + (unsigned)(((lane & 15) ^ fk(lane >> 4)) * 16);
  const unsigned vB1 = vB0 + 4 * (NL * 2) - 1024, vB2 = (vB0 ^ 128u) + 8 * (NL * 2) - 2048, vB3 = (vB0 ^ 128u) + 12 * (NL * 2) - 3072;
  const unsigned ldq = ldsb + (unsigned)wc * 4096u;
  const int aoff = lds_byte(wr * 64 + fr, fq * 8);
  const int q_ = (lane & 15) >> 2, p_ = lane & 3;
  const int f_ = (q_ & 1) | ((q_ & 2) << 1) | ((fq & 1) << 3);
  int boff0 = 4 * HTB + 256 * (8 * fq + q_) + 16 * ((4 * wc + p_) ^ f_) + 8 * (p_ & 1);
  int boff1 = boff0 ^ 8;
  asm volatile("" : "+v"(boff0), "+v"(boff1));
  const unsigned sA0 = (unsigned)(2 * (pm & 1)) * HTB, sB0 = (unsigned)pn * 512u;
  const unsigned sD0 = (unsigned)unit * 2048u;
  const unsigned vF = (unsigned)rank * FSRC + (unsigned)unit * FSTR;

#define SA(b, h) (((b) * 2 + (h)) * HTB)
#define SB(b, h) ((4 + (b) * 2 + (h)) * HTB)
#define PA_(b, h, kt) asm volatile("s_mov_b32 m0, %0\n\ts_nop 0\n\tbuffer_load_dwordx4 %1, %2, %3 offen sc0 sc1 lds" \
    :: "s"(ldq + SA(b, h)), "v"(vA), "s"(srdA), "s"(sA0 + (unsigned)(kt) * ASTEP + (h) * HTB) : "m0", "memory")
#define PB_(b, h, kt) asm volatile("s_mov_b32 m0, %0\n\ts_nop 0\n\tbuffer_load_dwordx4 %1, %2, %3 offen lds" \
    :: "s"(ldq + SB(b, h)), "v"(vB0), "s"(srdB), "s"(sB0 + (unsigned)(kt) * KSTEP + (h) * 256) : "m0", "memory")
#define PAN(h, kt, i) asm volatile("buffer_load_dwordx4 %0, %1, %2 offen offset:" #i " sc0 sc1 lds" \
    :: "v"(vA), "s"(srdA), "s"(sA0 + (unsigned)(kt) * ASTEP + (h) * HTB) : "memory")
#define PBN(h, kt, v, i) asm volatile("buffer_load_dwordx4 %0, %1, %2 offen offset:" #i " lds" \
    :: "v"(v), "s"(srdB), "s"(sB0 + (unsigned)(kt) * KSTEP + (h) * 256) : "memory")
#define STAGE_A(b, h, kt) do { PA_(b, h, kt); PAN(h, kt, 1024); PAN(h, kt, 2048); PAN(h, kt, 3072); } while (0)
#define STAGE_B(b, h, kt) do { PB_(b, h, kt); PBN(h, kt, vB1, 1024); PBN(h, kt, vB2, 2048); PBN(h, kt, vB3, 3072); } while (0)
#define TR(off) __builtin_amdgcn_ds_read_tr16_b64_v4i16((LAS s16x4*)(lds + (off)))
#define LDA2(dst, b, h, m0_) do { _Pragma("unroll") for (int m = m0_; m < m0_ + 2; ++m) _Pragma("unroll") for (int k = 0; k < 2; ++k) dst[m][k] = *(const LAS bf16x8*)(lds + SA(b, h) + aoff + m * 2048 + k * 1024); } while (0)
#define LDB1(dst, b, h, k) do { \
    { s16x4 lo = TR(boff0 + ((b) * 2 + (h)) * HTB + k * 8192), hi = TR(boff0 + ((b) * 2 + (h)) * HTB + k * 8192 + 1024); dst[0][k] = __builtin_shufflevector(lo, hi, 0, 1, 2, 3, 4, 5, 6, 7); } \
    { s16x4 lo = TR(boff1 + ((b) * 2 + (h)) * HTB + k * 8192), hi = TR(boff1 + ((b) * 2 + (h)) * HTB + k * 8192 + 1024); dst[1][k] = __builtin_shufflevector(lo, hi, 0, 1, 2, 3, 4, 5, 6, 7); } } while (0)
#define WAIT_V(n) asm volatile("s_waitcnt vmcnt(" #n ")" ::: "memory")
#define WAIT_L(n) do { asm volatile("" ::: "memory"); __builtin_amdgcn_s_waitcnt(0xC07F | ((n) << 8)); asm volatile("" ::: "memory"); } while (0)
#define BAR do { asm volatile("" ::: "memory"); __builtin_amdgcn_s_barrier(); asm volatile("" ::: "memory"); } while (0)
#define SCHED __builtin_amdgcn_sched_barrier(0)
#define MMA(ai, bj, At, Bt) do { _Pragma("unroll") for (int n = 0; n < 2; ++n) _Pragma("unroll") for (int m_ = 0; m_ < 4; ++m_) _Pragma("unroll") for (int k_ = 0; k_ < 2; ++k_) { \
    const int m = n ? 3 - m_ : m_, k = k_ ^ (m_ & 1); \
    acc[ai][bj][m][n] = __builtin_amdgcn_mfma_f32_16x16x32_bf16(Bt[n][k], At[m][k], acc[ai][bj][m][n], 0, 0, 0); SCHED; } } while (0)
#define MMA2(ai, At, B0, B1) do { SCHED; MMA(ai, 0, At, B0); MMA(ai, 1, At, B1); SCHED; } while (0)
#define MMA2W(ai, At, B0, B1, W) do { SCHED; MMA(ai, 0, At, B0); \
    _Pragma("unroll") for (int n = 0; n < 2; ++n) _Pragma("unroll") for (int m_ = 0; m_ < 4; ++m_) { if (n == 1 && m_ == WPOS) { W; SCHED; } _Pragma("unroll") for (int k_ = 0; k_ < 2; ++k_) { \
    const int m = n ? 3 - m_ : m_, k = k_ ^ (m_ & 1); \
    acc[ai][1][m][n] = __builtin_amdgcn_mfma_f32_16x16x32_bf16(B1[n][k], At[m][k], acc[ai][1][m][n], 0, 0, 0); SCHED; } } SCHED; } while (0)
#define PUBLISH(k, voff) do { __builtin_amdgcn_raw_buffer_store_b32(base + (unsigned)(k) + 1u, srdFd, voff, (k) < NKT ? (unsigned)((k) & 7) * FSLOT : OOB, 17); asm volatile("" ::: "memory"); } while (0)
#define PACE_UNTIL(ticks) do { while (__builtin_amdgcn_s_memrealtime() - t_start < (unsigned long long)(ticks)) __builtin_amdgcn_s_sleep(1); } while (0)

  f32x4 acc[2][2][4][2];
#pragma unroll
  for (int i = 0; i < 2; ++i)
#pragma unroll
    for (int j = 0; j < 2; ++j)
#pragma unroll
      for (int m = 0; m < 4; ++m)
#pragma unroll
        for (int n = 0; n < 2; ++n) acc[i][j][m][n] = (f32x4){0.f, 0.f, 0.f, 0.f};
  bf16x8 At[4][2], B0a[2][2], B0b[2][2], B1[2][2];
  if (!wr) {
#pragma unroll
    for (int k0 = 0; k0 < LEAD; k0 += 4) {
      const int k = k0 + wc;
      if (k < LEAD) {
        if (k0) { PLOAD(k, 0); PLOAD(k, 1); WAIT_V(0); } else WAIT_V(2);
        const u32x4 p0 = PREAD(0), p1 = PREAD(1);
        PACE_UNTIL(k * PACE);
        st128<17>(p0, srdD, (unsigned)lane * 16u, sD0 + (unsigned)k * ASTEP);
        st128<17>(p1, srdD, (unsigned)lane * 16u + 1024u, sD0 + (unsigned)k * ASTEP);
        WAIT_V(0);
        PUBLISH(k, lane == 0 ? vF : OOB);
      }
    }
    if (LEAD < 4) WAIT_V(0);
    WAIT_L(0);
    BAR;
    LDB1(B0a, 0, 0, 0); LDB1(B0a, 0, 0, 1);
    WAIT_L(0);
    BAR;
    unsigned vS = lane < 32 ? (unsigned)(wc * 512 + lane * 16) : OOB, vF0 = tid == 0 ? vF : OOB;
    asm volatile("" : "+v"(vS), "+v"(vF0));
    const int stgl = L_STG + wc * 1024 + lane * 16;
#define PUSH_RD(sl) (*(const LAS u32x4*)(lds + stgl + (sl) * 4096))
    u32x4 pv = PUSH_RD(0);
#define KTILE0(t, b, B0c, B0n) do { \
    PUBLISH((t) + LEAD - 1, vF0); \
    st128<17>(pv, srdD, vS, sD0 + (unsigned)((t) + LEAD) * ASTEP); \
    LDB1(B1, b, 1, 0); PB_(b, 0, (t) + 2); LDB1(B1, b, 1, 1); PBN(0, (t) + 2, vB1, 1024); \
    LDA2(At, b, 0, 0); PBN(0, (t) + 2, vB2, 2048); LDA2(At, b, 0, 2); PBN(0, (t) + 2, vB3, 3072); \
    WAIT_L(0); BAR; MMA2W(0, At, B0c, B1, WAIT_V(10)); BAR; SCHED; \
    XLOAD((t) + LEAD + 2, b); pv = PUSH_RD((b) ^ 1); \
    LDA2(At, b, 1, 0); PB_(b, 1, (t) + 2); LDA2(At, b, 1, 2); PBN(1, (t) + 2, vB1, 1024); \
    LDB1(B0n, (b) ^ 1, 0, 0); PBN(1, (t) + 2, vB2, 2048); LDB1(B0n, (b) ^ 1, 0, 1); PBN(1, (t) + 2, vB3, 3072); \
    WAIT_L(0); BAR; MMA2W(1, At, B0c, B1, WAIT_V(9)); BAR; SCHED; } while (0)
    for (int t = 0; t < NKT; t += 2) {
      KTILE0(t, 0, B0a, B0b);
      KTILE0(t + 1, 1, B0b, B0a);
    }
    BAR;
  } else {
    PACE_UNTIL(BDELAY);
    STAGE_B(0, 0, 0); STAGE_B(0, 1, 0); STAGE_B(1, 0, 1); STAGE_B(1, 1, 1);
    spin_flags(myflags, base + 1u, tmo, call, dead);
    STAGE_A(0, 0, 0); STAGE_A(0, 1, 0);
    POLL(1);
    WAIT_V(0);
    BAR;
    LDB1(B0a, 0, 0, 0); LDB1(B0a, 0, 0, 1);
    WAIT_L(0);
    BAR;
    BAR;
    const int chkl = L_POLL + (lane & 15) * 16;
#define FLAGS_OK(fv, kp) __all((int)((fv) - (base + (unsigned)(kp) + 1u)) >= 0)
#define CHECK(fv, kp) do { if (__builtin_expect((kp) < NKT && !FLAGS_OK(fv, kp), 0)) { \
        if (wid == 4) { WAIT_V(4); \
          if (!FLAGS_OK(*(volatile LAS unsigned*)(lds + chkl), kp)) { spin_flags(myflags + ((kp) & 7) * (FSLOT / 4), base + (unsigned)(kp) + 1u, tmo, call, dead); \
            if (lane == 0) *(volatile LAS unsigned*)(dead + 1) = (unsigned)(kp) + 1u; } } \
        else while (!FLAGS_OK(*(volatile LAS unsigned*)(lds + chkl), kp) && __builtin_amdgcn_readfirstlane(*(volatile LAS unsigned*)(dead + 1)) < (unsigned)(kp) + 1u) __builtin_amdgcn_s_sleep(1); \
        asm volatile("" ::: "memory"); } } while (0)
#define KTILE1(t, b, B0c, B0n) do { \
    { const unsigned fv_ = *(const LAS unsigned*)(lds + chkl); asm volatile("" ::: "memory"); \
      LDB1(B1, b, 1, 0); \
      CHECK(fv_, (t) + 1); } \
    PA_((b) ^ 1, 0, (t) + 1); LDB1(B1, b, 1, 1); PAN(0, (t) + 1, 1024); LDA2(At, b, 0, 0); \
    PAN(0, (t) + 1, 2048); LDA2(At, b, 0, 2); PAN(0, (t) + 1, 3072); \
    WAIT_V(4); WAIT_L(0); BAR; MMA2(0, At, B0c, B1); BAR; SCHED; \
    POLL((t) + 2); \
    PA_((b) ^ 1, 1, (t) + 1); LDA2(At, b, 1, 0); PAN(1, (t) + 1, 1024); LDA2(At, b, 1, 2); \
    PAN(1, (t) + 1, 2048); LDB1(B0n, (b) ^ 1, 0, 0); PAN(1, (t) + 1, 3072); LDB1(B0n, (b) ^ 1, 0, 1); \
    WAIT_V(5); WAIT_L(0); BAR; MMA2(1, At, B0c, B1); BAR; SCHED; } while (0)
    for (int t = 0; t < NKT; t += 2) {
      KTILE1(t, 0, B0a, B0b);
      KTILE1(t + 1, 1, B0b, B0a);
    }
  }
  {
    int tz = threadIdx.x; asm volatile("" : "+v"(tz));
    const int fr2 = tz & 15, fq2 = (tz >> 4) & 3;
    const bool odd = fq2 & 1;
    const __amdgpu_buffer_rsrc_t srdO = mk_rsrc(a.out, O_BYTES);
    const unsigned ooff = ((unsigned)(pm * BM + wr * 64 + fr2) * NL + (unsigned)(pn * BM + wc * 32 + 8 * fq2)) * 2u;
    if (__builtin_amdgcn_readfirstlane(*dead) != 0u) {
#pragma unroll
      for (int i = 0; i < 2; ++i)
#pragma unroll
        for (int j = 0; j < 2; ++j)
#pragma unroll
          for (int m = 0; m < 4; ++m)
#pragma unroll
            for (int n = 0; n < 2; ++n) acc[i][j][m][n] = (f32x4){__builtin_nanf(""), __builtin_nanf(""), __builtin_nanf(""), __builtin_nanf("")};
    }
#pragma unroll
    for (int ai = 0; ai < 2; ++ai)
#pragma unroll
      for (int m = 0; m < 4; ++m)
#pragma unroll
        for (int bj = 0; bj < 2; ++bj) {
          const f32x4 v0 = acc[ai][bj][m][0], v1 = acc[ai][bj][m][1];
          const unsigned a0 = gelu2((f32x2){v0[0], v0[1]}), a1 = gelu2((f32x2){v0[2], v0[3]});
          const unsigned b0 = gelu2((f32x2){v1[0], v1[1]}), b1 = gelu2((f32x2){v1[2], v1[3]});
          u32x4 w; w.x = odd ? b0 : a0; w.y = odd ? b1 : a1; w.z = odd ? a0 : b0; w.w = odd ? a1 : b1;
          st128<16>(w, srdO, ooff + bj * (HALF * 2), (unsigned)(ai * HALF + m * 16) * (NL * 2));
        }
    if (tz == 0) __hip_atomic_store((unsigned*)(ws + WS_CTR + (size_t)bid * 128), call, __ATOMIC_RELAXED, __HIP_MEMORY_SCOPE_AGENT);
  }
}
}

extern "C" void kernel_launch(int rank, int nranks, ncclComm_t comm,
                              hipStream_t stream,
                              void* const* d_in, const int* in_sizes, int n_in,
                              void* d_out, int out_size,
                              const int* in_dtypes, int out_dtype,
                              void* const* d_out_all,
                              void* const* d_ws_all, size_t ws_bytes) {
  using namespace agg;
  if (nranks != NR || (unsigned)rank >= (unsigned)NR || n_in < 2 || ws_bytes < WS_NEED || in_sizes[0] != MS * KT || in_sizes[1] != KT * NL || out_size != MT * NL) {
    fprintf(stderr, "kernel_launch: unexpected problem shape\n");
    return;
  }
  static unsigned attr_done = 0;
  int dev = 0; HIP_CHECK(hipGetDevice(&dev));
  if (!(attr_done >> (dev & 31) & 1u)) {
    HIP_CHECK(hipFuncSetAttribute((const void*)ag_gemm_kernel, hipFuncAttributeMaxDynamicSharedMemorySize, LDS_BYTES));
    attr_done |= 1u << (dev & 31);
  }
  Args a;
  a.x = (const char*)d_in[0]; a.W = (const char*)d_in[1]; a.out = (char*)d_out; a.rank = rank;
  for (int p = 0; p < NR; ++p) a.ws[p] = (char*)d_ws_all[(rank + p) & (NR - 1)];
  hipLaunchKernelGGL(ag_gemm_kernel, dim3(256), dim3(512), LDS_BYTES, stream, a);
  HIP_CHECK(hipGetLastError());
}
```
